```python
import numpy as np

kernel_name = "gpu_gemm_ar_m4096_k4096_n4096_bf16_reluq8"


ABSMAX_RTOL = 0.02

COMPUTE_DTYPE = "bfloat16"
EPILOGUE = "relu_quant_int8"
IN_QUANT = None
OUT_QUANT = "int8"
IN_DTYPES = {"x": "bfloat16", "w_mat": "bfloat16"}
WORLD_SIZE = 8

def _to_bf16(x):
    u = np.ascontiguousarray(x, dtype=np.float32).view(np.uint32)
    return ((u + 0x7FFF + ((u >> 16) & 1)) & 0xFFFF0000).view(np.float32)

def setup_inputs():

    x = _to_bf16(np.random.randn(4096, 4096).astype(np.float32))
    w_mat = _to_bf16((np.random.randn(4096, 4096) / np.sqrt(4096)).astype(np.float32))
    return {"x": x, "w_mat": w_mat}

def _quant_dequant(y, amax):
    scale = amax / 127.0
    q = np.clip(np.round(y / scale), -127.0, 127.0)
    return (q * scale).astype(np.float32)
def _epilogue(y):
    y = np.maximum(y, 0.0)
    return _quant_dequant(y, np.abs(y).max())

def reference(x, w_mat):
    y = (x @ w_mat).astype(np.float32)
    return _epilogue(y)
```

```cpp
#include <hip/hip_runtime.h>
#include <rccl/rccl.h>
#include <cstdio>
#include <cstdint>

#define HIP_CHECK(expr) do { hipError_t _e = (expr); if (_e != hipSuccess) { \
    fprintf(stderr, "HIP error %s at %s:%d\n", hipGetErrorString(_e), __FILE__, __LINE__); \
    return; } } while (0)

typedef unsigned int u32;
typedef unsigned long long u64;
typedef __attribute__((ext_vector_type(4))) u32 u32x4;
typedef __attribute__((ext_vector_type(2))) u32 u32x2;
typedef __attribute__((ext_vector_type(4))) float f32x4;
typedef __attribute__((ext_vector_type(8))) short bf16x8;
typedef __attribute__((address_space(3))) char lds_char;

namespace cfg {
constexpr int NR = 8;
constexpr int MDIM = 4096, NDIM = 4096, KSH = 512;
constexpr int NSTEP = 32;
constexpr int IMG = 16384;
constexpr int NBUF = 3;
constexpr int THREADS = 704;
constexpr int GRID = 256;
constexpr int NWV = 8;
#define N_PUSH 8
constexpr int NPUSH = N_PUSH;
#define O_DEPTH 2
constexpr int ODEPTH = O_DEPTH;
constexpr int NU = 320;
constexpr int NO = 128;
constexpr u32 OFF_A = 0;
constexpr u32 OFF_B = OFF_A + NSTEP * 16 * IMG;
constexpr u32 EXP_X = OFF_B + NSTEP * 16 * IMG;
constexpr u32 EXP_W = EXP_X + 256 * IMG;
constexpr u32 OFF_P = EXP_W + 256 * IMG;
constexpr u32 OFF_P2 = OFF_P + 128 * 32768;
constexpr u32 OFF_Q = OFF_P2 + 128 * 32768;
constexpr u32 QPITCH = 9216;
constexpr u32 OFF_CTL = OFF_Q + 256 * QPITCH;
constexpr u32 FLAG_A = OFF_CTL;
constexpr u32 FLAG_B = FLAG_A + 8192;
constexpr u32 PFLAG = FLAG_B + 4096;
constexpr u32 PF2 = PFLAG + 4096;
constexpr u32 EFLAG = PF2 + 2048;
constexpr u32 AMAX = EFLAG + 16384;
constexpr u32 OFLAG = AMAX + 8192;
constexpr u32 CNT = OFLAG + 8192;
constexpr u32 TMO = CNT + 128;
constexpr u32 TICK = TMO + 128;
constexpr u32 WS_USED = TICK + 2048;
constexpr u32 OFF_DBG = 40u << 20;
constexpr u32 OUT_BYTES = (u32)MDIM * NDIM;
constexpr u32 IN_BYTES = (u32)MDIM * KSH * 2;
constexpr int LDS_P = NBUF * 2 * IMG;
constexpr int LDS_CTL = LDS_P + 32768;
constexpr int LDS_BYTES = LDS_CTL + 256;
constexpr u64 DEADLINE = 30000000ull;
constexpr int AUX_SYS = 17;
constexpr int AUX_SC1 = 16;
}
using namespace cfg;

struct Params {
    char* ws[NR];
    char* out[NR];
    const char* x;
    const char* w;
    int rank;
};

#define STAMP(idx) do { } while (0)

__device__ __forceinline__ __amdgpu_buffer_rsrc_t mk_rsrc(const void* base, u32 bytes) {
    return __builtin_amdgcn_make_buffer_rsrc((void*)base, (short)0, (int)bytes, 0x00020000);
}
__device__ __forceinline__ int row_quarter(int r, int lq) { int ri = (r >> 1) & 1; return (r >> 2) ? ri + 2 * lq : 2 * ri + lq; }
__device__ __forceinline__ int col_quarter(int r, int lq) { int rj = r & 1; return (r >> 2) ? rj + 2 * lq : 2 * rj + lq; }
__device__ __forceinline__ int grow(int r, int rb) {
    int cls = r >> 2, half = (rb >> 3) ? cls : 1 - cls;
    return row_quarter(r, (rb >> 2) & 1) * 1024 + half * 512 + (rb & 3) * 128;
}
__device__ __forceinline__ int gcol(int r, int cb) { return col_quarter(r, cb >> 3) * 1024 + (cb & 7) * 128; }

__device__ __forceinline__ void pair_of(int rank, int r8, int cb, int& partner, int& tile) {
    const int u = row_quarter(rank, r8 >> 2), v = col_quarter(rank, cb >> 3);
    int lqr, lqc;
    if (rank < 4) { partner = 4 + ((u & 1) << 1) + (v & 1); lqr = u >> 1; lqc = v >> 1; }
    else { partner = ((u >> 1) << 1) + (v >> 1); lqr = u & 1; lqc = v & 1; }
    tile = (lqr * 4 + (r8 & 3)) * 16 + lqc * 8 + (cb & 7);
}

__device__ __forceinline__ bool timed_out(const Params& p, u32 seq, u64 t0) {
    u32* tmo = (u32*)(p.ws[p.rank] + TMO);
    if (__builtin_amdgcn_readfirstlane(__hip_atomic_load(tmo, __ATOMIC_RELAXED, __HIP_MEMORY_SCOPE_AGENT)) == seq ||
        __builtin_amdgcn_s_memrealtime() - t0 > DEADLINE) {
        __hip_atomic_store(tmo, seq, __ATOMIC_RELAXED, __HIP_MEMORY_SCOPE_AGENT);
        return true;
    }
    return false;
}

__device__ __forceinline__ float wave_max(float x) {
    u32 v = __float_as_uint(x);
    v = max(v, (u32)__builtin_amdgcn_update_dpp(0, (int)v, 0x111, 0xf, 0xf, true));
    v = max(v, (u32)__builtin_amdgcn_update_dpp(0, (int)v, 0x112, 0xf, 0xf, true));
    v = max(v, (u32)__builtin_amdgcn_update_dpp(0, (int)v, 0x114, 0xf, 0xf, true));
    v = max(v, (u32)__builtin_amdgcn_update_dpp(0, (int)v, 0x118, 0xf, 0xf, true));
    v = max(v, (u32)__builtin_amdgcn_update_dpp(0, (int)v, 0x142, 0xa, 0xf, true));
    v = max(v, (u32)__builtin_amdgcn_update_dpp(0, (int)v, 0x143, 0xc, 0xf, true));
    return __uint_as_float(__builtin_amdgcn_readlane(v, 63));
}
__device__ __forceinline__ u32 wave_prefix(u32 c, u32& total) {
    u32 off = 0; total = 0;
#pragma unroll
    for (int b = 0; b < 6; ++b) {
        const u64 m = __ballot((c >> b) & 1u);
        off += __builtin_amdgcn_mbcnt_hi((u32)(m >> 32), __builtin_amdgcn_mbcnt_lo((u32)m, 0u)) << b;
        total += (u32)__builtin_popcountll(m) << b;
    }
    return off;
}

struct Unit { u32 src; u32 dst; u32 flag; int isw; };

template <int NK>
__device__ __forceinline__ Unit decode(int u, int d, int sb, int sm) {
    constexpr int kbase = (NK == 5) ? 0 : 320;
    int kc, idx, g = 0, keep = 0;
    bool isw;
    if (u < 24 * NK) { kc = u / 24; idx = u % 24; isw = idx >= 16; idx &= 15; }
    else if (u < 48 * NK) { int v = u - 24 * NK; g = 1 + v / (8 * NK); v %= 8 * NK; kc = v >> 3; idx = v & 7; isw = true; }
    else { int v = u - 48 * NK; kc = v >> 4; idx = v & 15; isw = false; keep = 8; }
    const int half = idx & 1, s = sb + sm * kc;
    Unit r;
    r.isw = isw;
    if (isw) {
        const int cb = (idx >> 2) * 8 + g * 2 + ((idx >> 1) & 1);
        r.src = (u32)(((kbase + kc * 64) * NDIM + gcol(d, cb) + half * 64) * 2);
        r.dst = OFF_B + (u32)((s * 16 + cb) * IMG + half * 128);
        r.flag = FLAG_B + (u32)((cb >> 1) * 512 + s * 16 + (cb & 1) * 8 + half * 4);
    } else {
        const int rb = (idx >> 1) + keep;
        r.src = (u32)(((grow(d, rb) + half * 64) * KSH + kbase + kc * 64) * 2);
        r.dst = OFF_A + (u32)((s * 16 + rb) * IMG + half * 1024);
        r.flag = FLAG_A + (u32)(rb * 512 + s * 16 + half * 4);
    }
    return r;
}

__device__ __forceinline__ u32x4 mk_desc(const void* base, u32 bytes) {
    const u64 a = (u64)base;
    u32x4 d;
    d[0] = __builtin_amdgcn_readfirstlane((u32)a);
    d[1] = __builtin_amdgcn_readfirstlane((u32)(a >> 32) & 0xffffu);
    d[2] = bytes;
    d[3] = 0x00020000u;
    return d;
}
#define LD16_LDS(ldsa, voff, desc, soff) do { u32 keep_m0; \
    asm volatile("s_mov_b32 %0, m0\n\ts_mov_b32 m0, %1\n\ts_nop 0\n\t" \
                 "buffer_load_dwordx4 %2, %3, %4 offen lds\n\ts_mov_b32 m0, %0" \
                 : "=&s"(keep_m0) : "s"(ldsa), "v"(voff), "s"(desc), "s"(soff) : "memory"); } while (0)
#define ST16_SYS(src, voff, desc, soff) \
    asm volatile("buffer_store_dwordx4 %0, %1, %2, %3 offen sc0 sc1\n\ts_nop 1" :: "v"(src), "v"(voff), "s"(desc), "s"(soff) : "memory")
#define ST4_SYS(src, voff, desc) \
    asm volatile("buffer_store_dword %0, %1, %2, 0 offen sc0 sc1\n\ts_nop 1" :: "v"(src), "v"(voff), "s"(desc) : "memory")
constexpr u32 OOB = 0x80000000u;

__device__ __forceinline__ void unit_load(u32 ldsa, const Params& p, const Unit& un, int lane) {
    const u32x4 rs = mk_desc(un.isw ? p.w : p.x, IN_BYTES);
    const u32 voff = (u32)(((lane >> 3) << (un.isw ? 16 : 10)) + (lane & 7) * 16);
    asm volatile("s_nop 4" ::: "memory");
#pragma unroll
    for (int i = 0; i < 8; ++i) LD16_LDS(ldsa + i * 1024, voff, rs, un.src + i * 8192);
}

__device__ __forceinline__ void unit_store(const lds_char* buf, u32x4 rd, const Unit& un, int lane) {
    const u32 voff = (u32)((lane >> 3) * 2048 + (lane & 7) * 16);
    const u32 ds = un.isw ? 256u : 128u;
    const bool isw = un.isw != 0;
    const int roff = isw ? lane * 16 : (lane & 7) * 128 + (lane >> 3) * 16;
    u32x4 v[8];
#pragma unroll
    for (int i = 0; i < 8; ++i) v[i] = *(const __attribute__((address_space(3))) u32x4*)(buf + roff + i * 1024);
#pragma unroll
    for (int t = 0; t < 8; ++t) {
        u32x4 o;
#pragma unroll
        for (int m = 0; m < 4; ++m) {
            const u32 tr = __builtin_amdgcn_perm(v[2 * m + 1][t >> 1], v[2 * m][t >> 1], (t & 1) ? 0x07060302u : 0x05040100u);
            o[m] = isw ? tr : v[t][m];
        }
        ST16_SYS(o, voff, rd, un.dst + t * ds);
    }
}

#define LD16_LDS_SYS(ldsa, voff, desc, soff, imm) do { u32 keep_m0; \
    asm volatile("s_mov_b32 %0, m0\n\ts_mov_b32 m0, %1\n\ts_nop 0\n\t" \
                 "buffer_load_dwordx4 %2, %3, %4 offen offset:" #imm " sc0 sc1 lds\n\ts_mov_b32 m0, %0" \
                 : "=&s"(keep_m0) : "s"(ldsa), "v"(voff), "s"(desc), "s"(soff) : "memory"); } while (0)

__device__ __forceinline__ void step_map(int s, int d, int& sb, int& sm) {
    if ((s >> 2) != (d >> 2)) { sb = 5 + ((s - d) & 3); sm = 4; }
    else { sb = 17 + ((s - d - 1) & 3); sm = 3; }
}

__device__ __forceinline__ void copy_unit(const Params& p, lds_char* lds, u32x4 rd, const Unit& un, int lane) {
    unit_load(__builtin_amdgcn_readfirstlane((u32)(uintptr_t)lds), p, un, lane);
    asm volatile("s_waitcnt vmcnt(0)" ::: "memory");
    unit_store(lds, rd, un, lane);
    asm volatile("s_waitcnt vmcnt(0)" ::: "memory");
}

__device__ __forceinline__ void export_unit(const Params& p, lds_char* lds, int e, u32 seq, int lane) {
    const int rank = p.rank, isw = e >> 9, h2 = (e >> 3) & 63, kc8 = e & 7;
    Unit un;
    un.isw = isw;
    un.src = isw ? (u32)((kc8 * 64 * NDIM + h2 * 64) * 2) : (u32)((h2 * 64 * KSH + kc8 * 64) * 2);
    un.dst = (isw ? EXP_W + (h2 & 1) * 128 : EXP_X + (h2 & 1) * 1024) + (u32)((kc8 * 32 + (h2 >> 1)) * IMG);
    copy_unit(p, lds, mk_desc(p.ws[rank], WS_USED), un, lane);
    const int ccls = kc8 < 5 ? rank >> 2 : 1 - (rank >> 2), kcl = kc8 < 5 ? kc8 : kc8 - 5;
    const int q = h2 >> 4, sel = ccls ? q & 1 : q >> 1, lq = ccls ? q >> 1 : q & 1;
#pragma unroll
    for (int o = 0; o < 2; ++o) {
        const int c = ccls * 4 + (isw ? o * 2 + sel : sel * 2 + o);
        u32 fo = EFLAG + (u32)(rank * 2048 + (kcl * 64 + isw * 32 + lq * 16 + (h2 & 15)) * 4);
        if (c == rank) {
            const int b8 = (h2 >> 1) & 7;
            if (isw) { const int cb = lq * 8 + b8; fo = FLAG_B + (u32)((cb >> 1) * 512 + kcl * 16 + (cb & 1) * 8 + (h2 & 1) * 4); }
            else { const int rb = ((b8 >> 2) == ccls ? 8 : 0) + lq * 4 + (b8 & 3); fo = FLAG_A + (u32)(rb * 512 + kcl * 16 + (h2 & 1) * 4); }
        }
        const u32x4 rc = mk_desc(p.ws[c], WS_USED);
        asm volatile("s_nop 4" ::: "memory");
        ST4_SYS(seq, lane == 0 ? fo : OOB, rc);
    }
}

struct PUnit { u32 src; u32 dst; u32 flag; u32 wait; };
template <int NK>
__device__ __forceinline__ PUnit pdecode(int u, int rank, int d, int sb, int sm) {
    PUnit r;
    if (NK == 3 && u >= 64 * NK) {
        const int v = u - 64 * NK, g = v >> 5, tl = (v >> 2) & 7, q = v & 3, off = (tl & 3) * 16 + 2 * g + (tl >> 2);
        const int tile = ((d >> 1) & 1) * 64 + (d & 1) * 8 + off, ptile = ((rank >> 1) & 1) * 64 + (rank & 1) * 8 + off;
        r.src = OFF_P + (u32)(ptile * 32768 + q * 8192);
        r.dst = OFF_P2 + (u32)(tile * 32768 + q * 8192);
        r.flag = PF2 + (u32)(tile * 16 + q * 4);
        r.wait = (u32)(v >> 2) * 2;
        return r;
    }
    int kc, idx, g = 0, keep = 0;
    bool isw;
    if (u < 24 * NK) { kc = u / 24; idx = u % 24; isw = idx >= 16; idx &= 15; }
    else if (u < 48 * NK) { int v = u - 24 * NK; g = 1 + v / (8 * NK); v %= 8 * NK; kc = v >> 3; idx = v & 7; isw = true; }
    else { int v = u - 48 * NK; kc = v >> 4; idx = v & 15; isw = false; keep = 8; }
    const int half = idx & 1, s = sb + sm * kc, kc8 = kc + (NK == 5 ? 0 : 5);
    r.wait = OOB;
    if (isw) {
        const int cb = (idx >> 2) * 8 + g * 2 + ((idx >> 1) & 1);
        r.src = EXP_W + (u32)((kc8 * 32 + (gcol(rank, cb) >> 7)) * IMG + half * 8192);
        r.dst = OFF_B + (u32)((s * 16 + cb) * IMG + half * 8192);
        r.flag = FLAG_B + (u32)((cb >> 1) * 512 + s * 16 + (cb & 1) * 8 + half * 4);
    } else {
        const int rb = (idx >> 1) + keep;
        r.src = EXP_X + (u32)((kc8 * 32 + (grow(rank, rb) >> 7)) * IMG + half * 8192);
        r.dst = OFF_A + (u32)((s * 16 + rb) * IMG + half * 8192);
        r.flag = FLAG_A + (u32)(rb * 512 + s * 16 + half * 4);
    }
    return r;
}

__device__ __forceinline__ void wait_words(const Params& p, __amdgpu_buffer_rsrc_t rw, u32 off, int mask, u32 seq, int lane) {
    const u32 o = off + (u32)(lane & mask) * 4;
    if (__all(__builtin_amdgcn_raw_buffer_load_b32(rw, o, 0, AUX_SYS) >= seq)) return;
    const u64 t0 = __builtin_amdgcn_s_memrealtime();
    while (!__all(__builtin_amdgcn_raw_buffer_load_b32(rw, o, 0, AUX_SYS) >= seq)) {
        if (timed_out(p, seq, t0)) break;
        __builtin_amdgcn_s_sleep(1);
    }
}

__device__ __forceinline__ void pull8k(u32 ldsa, u32x4 rs, u32 src, int lane) {
    const u32 voff = (u32)lane * 16;
    ldsa = __builtin_amdgcn_readfirstlane(ldsa);
    src = __builtin_amdgcn_readfirstlane(src);
    asm volatile("s_nop 4" :: "s"(ldsa), "s"(src) : "memory");
    LD16_LDS_SYS(ldsa, voff, rs, src, 0); LD16_LDS_SYS(ldsa, voff, rs, src, 1024);
    LD16_LDS_SYS(ldsa, voff, rs, src, 2048); LD16_LDS_SYS(ldsa, voff, rs, src, 3072);
    LD16_LDS_SYS(ldsa + 4096, voff, rs, src + 4096, 0); LD16_LDS_SYS(ldsa + 4096, voff, rs, src + 4096, 1024);
    LD16_LDS_SYS(ldsa + 4096, voff, rs, src + 4096, 2048); LD16_LDS_SYS(ldsa + 4096, voff, rs, src + 4096, 3072);
}

__device__ __forceinline__ void push_unit(const Params& p, lds_char* lds, int d, int u, u32 seq, int lane) {
    int sb, sm;
    step_map(p.rank, d, sb, sm);
    const Unit un = (d >> 2) == (p.rank >> 2) ? decode<5>(u, d, sb, sm) : decode<3>(u, d, sb, sm);
    const u32x4 rp = mk_desc(p.ws[d], WS_USED);
    copy_unit(p, lds, rp, un, lane);
    ST4_SYS(seq, lane == 0 ? un.flag : OOB, rp);
}

template <int NK>
__device__ __forceinline__ void mover(const Params& p, lds_char* lds, int d, int wv, u32 seq, int lane) {
    const int rank = p.rank;
    const __amdgpu_buffer_rsrc_t rw = mk_rsrc(p.ws[rank], WS_USED);
    const u32x4 rl = mk_desc(p.ws[rank], WS_USED), rp = mk_desc(p.ws[d], WS_USED);
    const u32 ldsa = __builtin_amdgcn_readfirstlane((u32)(uintptr_t)lds);
    int sb, sm;
    step_map(d, rank, sb, sm);
#pragma unroll
    for (int i = 0; i < NK; ++i) wait_words(p, rw, EFLAG + (u32)(d * 2048 + i * 256), 63, seq, lane);
    const u32 pfo = PFLAG + (u32)((((d >> 1) & 1) * 64 + (d & 1) * 8 + ((lane >> 1) & 3) * 16 + 2 * (lane >> 4) + ((lane >> 3) & 1)) * 32 + (lane & 1) * 16);
    u64 pmask = 0;
    const u32 tbase = (seq - 1u) * (u32)(NU - NPUSH - NWV) - (u32)(NPUSH + 2 * NWV);
    int u = NPUSH + wv, u1 = NPUSH + NWV + wv;
    PUnit cur = pdecode<NK>(u, rank, d, sb, sm);
    u32 tk = (u32)__builtin_amdgcn_raw_ptr_buffer_atomic_add_i32(1, rw, lane == 0 ? TICK + d * 256 : OOB, 0, AUX_SC1);
    pull8k(ldsa, rp, cur.src, lane);
    u32x4 pf = {0u, 0u, 0u, 0u};
    u32 pflag = OOB;
    int par = 0;
    for (;;) {
        asm volatile("s_waitcnt vmcnt(0)" ::: "memory");
        const u32 n = __builtin_amdgcn_readfirstlane(tk) - tbase;
        if (NK == 3) pmask |= __ballot(pf[0] >= seq && pf[1] >= seq && pf[2] >= seq && pf[3] >= seq);
        asm volatile("" :: "s"(n), "s"(pmask) : "memory");
        const int u2 = (u1 < NU && n < (u32)NU) ? (int)n : NU;
        const PUnit nxt = pdecode<NK>(u1 < NU ? u1 : u, rank, d, sb, sm);
        const bool now = u1 < NU && !(NK == 3 && nxt.wait != OOB && ((pmask >> nxt.wait) & 3) != 3);
        if (now) {
            tk = (u32)__builtin_amdgcn_raw_ptr_buffer_atomic_add_i32(1, rw, (u2 < NU && lane == 0) ? TICK + d * 256 : OOB, 0, AUX_SC1);
            pull8k(ldsa + (par ^ 8192), rp, nxt.src, lane);
            if (NK == 3) pf = __builtin_amdgcn_raw_buffer_load_b128(rw, pfo, 0, AUX_SYS);
        }
        ST4_SYS(seq, pflag, rl);
        u32x4 v[8];
#pragma unroll
        for (int i = 0; i < 8; ++i) v[i] = *(const __attribute__((address_space(3))) u32x4*)(lds + par + lane * 16 + i * 1024);
#pragma unroll
        for (int i = 0; i < 8; ++i) ST16_SYS(v[i], (u32)lane * 16, rl, cur.dst + i * 1024);
        pflag = lane == 0 ? cur.flag : OOB;
        if (!now) {
            asm volatile("s_waitcnt vmcnt(0)" ::: "memory");
            ST4_SYS(seq, pflag, rl);
            pflag = OOB;
            if (u1 >= NU) break;
            const u64 t0 = __builtin_amdgcn_s_memrealtime();
            for (;;) {
                const u32x4 f = __builtin_amdgcn_raw_buffer_load_b128(rw, pfo, 0, AUX_SYS);
                pmask = __ballot(f[0] >= seq && f[1] >= seq && f[2] >= seq && f[3] >= seq);
                if (((pmask >> nxt.wait) & 3) == 3 || timed_out(p, seq, t0)) break;
                __builtin_amdgcn_s_sleep(1);
            }
            tk = (u32)__builtin_amdgcn_raw_ptr_buffer_atomic_add_i32(1, rw, (u2 < NU && lane == 0) ? TICK + d * 256 : OOB, 0, AUX_SC1);
            pull8k(ldsa + (par ^ 8192), rp, nxt.src, lane);
        }
        cur = nxt; u = u1; u1 = u2; par ^= 8192;
    }
}

constexpr int OB_IN = 16384, OB_PITCH = 9472, OB_LUT = OB_IN + 8 * OB_PITCH, OB_INFO = OB_LUT + 64, OB_IMG = OB_INFO + 64;
typedef volatile __attribute__((address_space(3))) u32 lds_vu32;

__device__ __forceinline__ void out_request(const Params& p, lds_char* lds, int d, int wv, u32 seq, int lane) {
    const int rank = p.rank;
    const __amdgpu_buffer_rsrc_t rw = mk_rsrc(p.ws[rank], WS_USED);
    const u32x4 rp = mk_desc(p.ws[d], WS_USED);
    const u32 ldsa = __builtin_amdgcn_readfirstlane((u32)(uintptr_t)lds) + OB_IN;
    lds_vu32* info = (lds_vu32*)(lds + OB_INFO);
    u32 len0 = 0, len1 = 0;
    auto wait_tiles = [&](u64 need) {
        const u64 t0 = __builtin_amdgcn_s_memrealtime();
        for (;;) {
            const u32x4 f = __builtin_amdgcn_raw_buffer_load_b128(rw, OFLAG + (u32)(d * 1024 + lane * 16), 0, AUX_SYS);
            len0 = f[0]; len1 = f[2];
            if ((__ballot(f[1] == seq && f[3] == seq) & need) == need || timed_out(p, seq, t0)) break;
            __builtin_amdgcn_s_sleep(1);
        }
    };
    auto plane_len = [&](int i) -> u32 {
        const int tile = wv + NWV * (i >> 1);
        const u32 v = __builtin_amdgcn_readlane((tile & 1) ? len1 : len0, tile >> 1);
        const u32 L = (v >> ((i & 1) * 16)) & 0xffffu;
        return L < 1024u ? L : 1024u;
    };
    constexpr int NH = 2 * NO / NWV;
    auto request = [&](int i) {
        const u32 bytes = i < NH ? 1040u + 7u * plane_len(i) : 0u;
        const u32 src = __builtin_amdgcn_readfirstlane(OFF_Q + (u32)((wv + NWV * (i >> 1)) * 2 + (i & 1)) * QPITCH);
        const u32 la = __builtin_amdgcn_readfirstlane(ldsa + (i & 7) * OB_PITCH);
        asm volatile("s_nop 4" :: "s"(la), "s"(src) : "memory");
#pragma unroll
        for (int o = 0; o < 9; ++o) {
            const u32 voff = o * 1024 + (u32)lane * 16 < bytes ? (u32)lane * 16 : OOB;
            LD16_LDS_SYS(la + o * 1024, voff, rp, src + o * 1024, 0);
        }
    };
    wait_tiles(1ull << (wv >> 1));
    request(0); request(1);
    wait_tiles(~0ull);
    const u32 bad = __builtin_amdgcn_readfirstlane(
        __hip_atomic_load((u32*)(p.ws[rank] + TMO), __ATOMIC_RELAXED, __HIP_MEMORY_SCOPE_AGENT)) == seq ? 0x10000u : 0u;
    for (int i = 2; i < ODEPTH; ++i) request(i);
    for (int i = 0; i < NH; ++i) {
        asm volatile("s_waitcnt vmcnt(%0)" :: "n"(9 * (ODEPTH - 1)) : "memory");
        request(i + ODEPTH);
        const u32 L = plane_len(i);
        if ((u32)lane * 16 < L) {
            const lds_char* pl = lds + OB_IN + (i & 7) * OB_PITCH + 1040 + lane * 16;
            u32x4 p7 = {0u, 0u, 0u, 0u};
#pragma unroll
            for (int j = 0; j < 7; ++j)
                p7 |= ((*(const __attribute__((address_space(3))) u32x4*)(pl + j * L) >> 7) & 0x01010101u) << j;
            *(__attribute__((address_space(3))) u32x4*)(pl + 7 * L) = p7;
        }
        if (i & 1) {
            if (lane == 0) info[(i >> 1) & 1] = (u32)(wv + NWV * (i >> 1)) | bad | ((u32)((i >> 1) & 3) << 20) | (i == NH - 1 ? 0x20000u : 0u);
            asm volatile("s_waitcnt lgkmcnt(0)" ::: "memory");
            __builtin_amdgcn_s_barrier();
        }
    }
    asm volatile("s_waitcnt vmcnt(0)" ::: "memory");
}

__device__ __forceinline__ void out_expand(const Params& p, lds_char* lds, int d, int e, int lane) {
    const int k = e & 3, h = e >> 2;
    const u32x4 ro = mk_desc(p.out[p.rank], OUT_BYTES);
    lds_vu32* info = (lds_vu32*)(lds + OB_INFO);
    if (e == 0 && lane < 16) {
        u32 sel = 0, c = 0;
        for (int i = 0; i < 4; ++i) { sel |= (((lane >> i) & 1) ? c++ : 0x0cu) << (8 * i); }
        *(__attribute__((address_space(3))) u32*)(lds + OB_LUT + lane * 4) = sel;
    }
    lds_char* img = lds + OB_IMG + e * 2304;
    const int fr = lane & 15, fq = lane >> 4, sh = (k & 1) * 16;
    const u32 ovoff = (u32)((e * 16 + (lane >> 3)) * NDIM + (lane & 7) * 16);
    for (int par = 0;; par ^= 1) {
        asm volatile("s_waitcnt lgkmcnt(0)" ::: "memory");
        __builtin_amdgcn_s_barrier();
        asm volatile("" ::: "memory");
        const u32 inf = __builtin_amdgcn_readfirstlane(info[par]);
        const int tile = inf & 0xff;
        const lds_char* in = lds + OB_IN + ((inf >> 20) * 2 + h) * OB_PITCH;
        u32 o[2][4];
#pragma unroll
        for (int wn = 0; wn < 2; ++wn) {
            const int pw = (k >> 1) * 2 + wn;
            const u32 m = *(const __attribute__((address_space(3))) u32*)(in + 16 + pw * 256 + lane * 4);
            u32 tot;
            u32 pos = *(const __attribute__((address_space(3))) u32*)(in + pw * 4) + wave_prefix(__builtin_popcount(m), tot);
            pos = (pos + __builtin_popcount(m & ((1u << sh) - 1u))) & 0x3fffu;
#pragma unroll
            for (int j = 0; j < 4; ++j) {
                const u32 m4 = (m >> (sh + 4 * j)) & 15u;
                const __attribute__((address_space(3))) u32* sp = (const __attribute__((address_space(3))) u32*)(in + 1040 + (pos & ~3u));
                const u32 by = __builtin_amdgcn_alignbyte(sp[1], sp[0], pos & 3u);
                o[wn][j] = __builtin_amdgcn_perm(0u, by, *(const __attribute__((address_space(3))) u32*)(lds + OB_LUT + m4 * 4)) & 0x7f7f7f7fu;
                pos += __builtin_popcount(m4);
            }
        }
#pragma unroll
        for (int j = 0; j < 4; ++j)
            *(__attribute__((address_space(3))) u32x2*)(img + (fq * 4 + j) * 144 + fr * 8) = u32x2{o[0][j], o[1][j]};
        const u32 obase = (u32)(grow(d, 8 + (tile >> 4)) * NDIM + gcol(d, tile & 15));
#pragma unroll
        for (int i = 0; i < 2; ++i) {
            u32x4 v = *(const __attribute__((address_space(3))) u32x4*)(img + (i * 8 + (lane >> 3)) * 144 + (lane & 7) * 16);
            if (inf & 0x10000u) v = u32x4{0x80808080u, 0x80808080u, 0x80808080u, 0x80808080u};
            ST16_SYS(v, ovoff, ro, obase + i * 8 * NDIM);
        }
        if (inf & 0x20000u) break;
    }
}

typedef volatile __attribute__((address_space(3))) int lds_int;
constexpr int SLOT = 2 * IMG;

__device__ __forceinline__ int step_of(int t, bool g0) {
    if (!g0 || t < 5 || t >= 26) return t;
    const int q = (t - 5) / 7, r = (t - 5) % 7;
    return r < 4 ? 5 + 4 * q + r : 13 + 3 * q + r;
}

__device__ __forceinline__ void glds4k(const char* g, lds_char* l) {
    const __attribute__((address_space(1))) void* gp = (const __attribute__((address_space(1))) void*)g;
    __attribute__((address_space(3))) void* lp = (__attribute__((address_space(3))) void*)l;
    __builtin_amdgcn_global_load_lds(gp, lp, 16, 0, 0);
    __builtin_amdgcn_global_load_lds(gp, lp, 16, 1024, 0);
    __builtin_amdgcn_global_load_lds(gp, lp, 16, 2048, 0);
    __builtin_amdgcn_global_load_lds(gp, lp, 16, 3072, 0);
}

__device__ __forceinline__ void gemm_poller(const Params& p, lds_char* lds, int rb, int cb, u32 seq, int lane) {
    const bool keep = rb >= 8, g0 = !keep && (cb & 6) == 0;
    const __amdgpu_buffer_rsrc_t rw = mk_rsrc(p.ws[p.rank], WS_USED);
    const u32 foff = (lane < 32 ? FLAG_A + rb * 512 : FLAG_B + (cb >> 1) * 512 + (cb & 1) * 8) + step_of(lane & 31, g0) * 16;
    lds_int* lready = (lds_int*)(lds + LDS_CTL) + keep;
    const char* psrc = p.ws[p.rank] + OFF_P2 + ((rb & 7) * 16 + cb) * 32768 + lane * 16;
    const u32 pfoff = PF2 + ((rb & 7) * 16 + cb) * 16 + (lane & 3) * 4;
    bool pdone = !keep;
    auto poll = [&]() -> int {
        const u32x2 f = __builtin_amdgcn_raw_buffer_load_b64(rw, foff, 0, AUX_SYS);
        const u64 m = __ballot(f[0] >= seq && f[1] >= seq);
        const u32 both = (u32)m & (u32)(m >> 32);
        return both == 0xffffffffu ? 32 : __builtin_ctz(~both);
    };
    u32 pfv = 0;
    auto pload = [&]() { pfv = __builtin_amdgcn_raw_buffer_load_b32(rw, pdone ? OOB : pfoff, 0, AUX_SYS); };
    auto pstep = [&]() {
        if (!pdone && __all(pfv >= seq)) {
#pragma unroll
            for (int i = 0; i < 8; ++i) glds4k(psrc + i * 4096, lds + LDS_P + i * 4096);
            pdone = true;
        }
    };
    int ready = 0;
    for (int t = 0; t < NSTEP; ++t) {
        if (ready < NSTEP && ready < t + NBUF) {
            int r = poll();
            if (r <= t) {
                const u64 t0 = __builtin_amdgcn_s_memrealtime();
                int spins = 0;
                do {
                    pload();
                    r = poll();
                    if (r <= t) {
                        pstep();
                        if ((++spins & 63) == 0 && timed_out(p, seq, t0)) r = NSTEP;
                    }
                } while (r <= t);
            }
            if (r > ready) {
                ready = r;
                if (lane == 0) *lready = r;
                asm volatile("s_waitcnt lgkmcnt(0)" ::: "memory");
            }
        }
        __builtin_amdgcn_s_barrier();
    }
    if (!pdone) {
        const u64 t0 = __builtin_amdgcn_s_memrealtime();
        for (int spins = 1;; ++spins) {
            pload();
            pstep();
            if (pdone || ((spins & 63) == 0 && timed_out(p, seq, t0))) break;
        }
    }
    asm volatile("s_waitcnt vmcnt(0)" ::: "memory");
    __builtin_amdgcn_s_barrier();
}

__device__ __forceinline__ void gemm_loader(const Params& p, lds_char* lds, int rb, int cb, int lw, int lane) {
    const bool keep = rb >= 8, g0 = !keep && (cb & 6) == 0;
    const char* wsl = p.ws[p.rank];
    lds_int* lready = (lds_int*)(lds + LDS_CTL) + keep;
    const char* ga = wsl + OFF_A + rb * IMG + lw * 8192 + lane * 16;
    const char* gb = wsl + OFF_B + cb * IMG + lw * 8192 + lane * 16;
    const char* ea = wsl + EXP_X + (grow(p.rank, rb) >> 7) * IMG + lw * 8192 + lane * 16;
    const char* eb = wsl + EXP_W + (gcol(p.rank, cb) >> 7) * IMG + lw * 8192 + lane * 16;
    int issued = 0, ready = 0;
    auto issue_upto = [&](int lim) {
        if (issued >= ready && ready < NSTEP) ready = __builtin_amdgcn_readfirstlane(*lready);
        if (lim > ready) lim = ready;
        for (; issued < lim; ++issued) {
            lds_char* dst = lds + (issued % NBUF) * SLOT + lw * 8192;
            const bool own = issued < 5;
            const size_t so = own ? (size_t)issued * (32 * IMG) : (size_t)step_of(issued, g0) * (16 * IMG);
            const char* sa = (own ? ea : ga) + so;
            const char* sb = (own ? eb : gb) + so;
            glds4k(sa, dst); glds4k(sa + 4096, dst + 4096);
            glds4k(sb, dst + IMG); glds4k(sb + 4096, dst + IMG + 4096);
        }
    };
    for (int t = 0; t < NSTEP; ++t) {
        while (issued <= t) {
            issue_upto(t + NBUF - 1);
            if (issued <= t) __builtin_amdgcn_s_sleep(1);
        }
        if (issued - t == 1) asm volatile("s_waitcnt vmcnt(0)" ::: "memory");
        else asm volatile("s_waitcnt vmcnt(16)" ::: "memory");
        __builtin_amdgcn_s_barrier();
        asm volatile("" ::: "memory");
        issue_upto(t + NBUF);
    }
    __builtin_amdgcn_s_barrier();
}

__device__ __forceinline__ void frag_mma(f32x4 (&acc)[2][4], const bf16x8 (&fa)[2], const bf16x8 (&fb)[4]) {
#pragma unroll
    for (int m = 0; m < 2; ++m)
#pragma unroll
        for (int n = 0; n < 4; ++n)
            acc[m][n] = __builtin_amdgcn_mfma_f32_16x16x32_bf16(fa[m], fb[n], acc[m][n], 0, 0, 0);
}
__device__ __forceinline__ void frag_read(bf16x8 (&fa)[2], bf16x8 (&fb)[4], const lds_char* a, const lds_char* b) {
#pragma unroll
    for (int m = 0; m < 2; ++m) fa[m] = *(const __attribute__((address_space(3))) bf16x8*)(a + m * 256);
#pragma unroll
    for (int n = 0; n < 4; ++n) fb[n] = *(const __attribute__((address_space(3))) bf16x8*)(b + n * 256);
}
__device__ __forceinline__ void interleave() {
#pragma unroll
    for (int i = 0; i < 6; ++i) {
        __builtin_amdgcn_sched_group_barrier(0x100, 1, 0);
        __builtin_amdgcn_sched_group_barrier(0x8, 1, 0);
    }
    __builtin_amdgcn_sched_barrier(0);
}

__device__ __forceinline__ void gemm_compute(lds_char* lds, int wave, int lane, f32x4 (&acc)[2][4]) {
    const int fr = lane & 15, fq = lane >> 4;
    const int aoff = fq * 2048 + ((wave >> 1) * 32 + fr) * 16;
    const int boff = IMG + fq * 2048 + (wave & 1) * 1024 + fr * 16;
    bf16x8 a0[2], b0[4], a1[2], b1[4];
#pragma unroll
    for (int m = 0; m < 2; ++m) {
#pragma unroll
        for (int n = 0; n < 4; ++n) acc[m][n] = f32x4{0.f, 0.f, 0.f, 0.f};
        a1[m] = bf16x8{0, 0, 0, 0, 0, 0, 0, 0};
    }
#pragma unroll
    for (int i = 0; i < 4; ++i) b1[i] = bf16x8{0, 0, 0, 0, 0, 0, 0, 0};
    for (int t = 0; t < NSTEP; ++t) {
        asm volatile("s_waitcnt lgkmcnt(0)" ::: "memory");
        __builtin_amdgcn_s_barrier();
        asm volatile("" ::: "memory");
        const lds_char* buf = lds + (t % NBUF) * SLOT;
        __builtin_amdgcn_sched_barrier(0);
        frag_read(a0, b0, buf + aoff, buf + boff);
        frag_mma(acc, a1, b1);
        interleave();
        frag_read(a1, b1, buf + aoff + 8192, buf + boff + 8192);
        frag_mma(acc, a0, b0);
        interleave();
    }
    frag_mma(acc, a1, b1);
    asm volatile("s_waitcnt lgkmcnt(0)" ::: "memory");
    __builtin_amdgcn_s_barrier();
    asm volatile("" ::: "memory");
}

__device__ __forceinline__ u32 pack_bf16(float lo, float hi) {
    typedef __attribute__((ext_vector_type(2))) __bf16 bf2;
    bf2 r; r[0] = (__bf16)lo; r[1] = (__bf16)hi;
    return __builtin_bit_cast(u32, r);
}

__device__ __forceinline__ u32x4 pack8(const f32x4& a, const f32x4& b) {
    u32x4 o;
    o[0] = pack_bf16(a[0], a[1]); o[1] = pack_bf16(a[2], a[3]);
    o[2] = pack_bf16(b[0], b[1]); o[3] = pack_bf16(b[2], b[3]);
    return o;
}

__global__ void __launch_bounds__(THREADS, 3) fused_kernel(Params p) {
    extern __shared__ __attribute__((aligned(16))) char smem[];
    lds_char* lds = (lds_char*)smem;
    const int tid = threadIdx.x, lane = tid & 63;
    const int wave = __builtin_amdgcn_readfirstlane(tid >> 6);
    const int rank = p.rank;
    const int xcd = blockIdx.x & 7, slot = blockIdx.x >> 3;
    char* wsl = p.ws[rank];
    const u32 seq = __builtin_amdgcn_readfirstlane(
        __hip_atomic_load((u32*)(wsl + CNT), __ATOMIC_RELAXED, __HIP_MEMORY_SCOPE_AGENT)) + 1u
        ;
    STAMP(0);

    if (slot < 8) {
        const int d = (rank + slot) & 7, wv = (xcd + slot) & 7;
        if (slot == 0 || wave > 8) return;
        if (wave) { out_expand(p, lds, d, wave - 1, lane); return; }
        if ((d >> 2) == (rank >> 2)) mover<5>(p, lds, d, wv, seq, lane);
        else mover<3>(p, lds, d, wv, seq, lane);
        out_request(p, lds, d, wv, seq, lane);
        return;
    }

    if (tid < 2) ((lds_int*)(lds + LDS_CTL))[tid] = 0;
    __syncthreads();
    const int srb = (xcd & 1) * 4 + (slot & 3);
    const int scb = (xcd >> 2) * 8 + (slot < 16 ? 4 : 0) + ((slot >> 2) & 1) * 2 + ((xcd >> 1) & 1);
    const int rbk = (xcd & 1) * 4 + (slot & 3);
    const int cb = (xcd >> 1) * 4 + ((slot >> 2) & 3);
    const int tile = rbk * 16 + cb;
    const int first = slot < 24 ? 0 : 1, last = slot < 16 ? 0 : 1;
    if (wave >= 8) {
        for (int pass = first; pass <= last; ++pass) {
            const int rb = pass ? 8 + rbk : srb, c = pass ? cb : scb;
            if (wave == 10) gemm_poller(p, lds, rb, c, seq, lane);
            else gemm_loader(p, lds, rb, c, wave - 8, lane);
        }
        return;
    }
    if (slot >= 16) export_unit(p, lds + 4 * IMG + wave * 8192, wave * 128 + tile, seq, lane);
    else if (wave * 64 + (slot - 8) * 8 + xcd < 7 * NPUSH) {
        const int id = wave * 64 + (slot - 8) * 8 + xcd;
        push_unit(p, lds + 4 * IMG + wave * 8192, (rank + 1 + id % 7) & 7, id / 7, seq, lane);
    }
    STAMP(6);
    const __amdgpu_buffer_rsrc_t rw = mk_rsrc(wsl, WS_USED);
    f32x4 acc[2][4];
    for (int pass = first;; ++pass) {
        gemm_compute(lds, wave, lane, acc);
        if (pass) break;
        STAMP(7);
        int partner, ptile;
        pair_of(rank, srb, scb, partner, ptile);
        const __amdgpu_buffer_rsrc_t rd = mk_rsrc(p.ws[partner], WS_USED);
#pragma unroll
        for (int m = 0; m < 2; ++m)
#pragma unroll
            for (int h = 0; h < 2; ++h)
                __builtin_amdgcn_raw_buffer_store_b128(pack8(acc[m][2 * h], acc[m][2 * h + 1]), rw, tid * 16,
                                                       OFF_P + (srb * 16 + scb) * 32768 + (m * 2 + h) * 8192, AUX_SYS);
        asm volatile("s_waitcnt vmcnt(0)" ::: "memory");
        if (lane == 0) __builtin_amdgcn_raw_buffer_store_b32(seq, rd, PFLAG + ptile * 32 + wave * 4, 0, AUX_SYS);
        STAMP(8);
        if (!last) return;
    }
    STAMP(2);
    float mx = 0.f;
#pragma unroll
    for (int m = 0; m < 2; ++m)
#pragma unroll
        for (int h = 0; h < 2; ++h) {
            const u32x4 o = *(const __attribute__((address_space(3))) u32x4*)(lds + LDS_P + (m * 2 + h) * 8192 + tid * 16);
#pragma unroll
            for (int q = 0; q < 4; ++q) {
                float lo = __builtin_bit_cast(float, o[q] << 16), hi = __builtin_bit_cast(float, o[q] & 0xffff0000u);
                f32x4& a = acc[m][2 * h + (q >> 1)];
                a[(q & 1) * 2] = fmaxf(a[(q & 1) * 2] + lo, 0.f);
                a[(q & 1) * 2 + 1] = fmaxf(a[(q & 1) * 2 + 1] + hi, 0.f);
                mx = fmaxf(mx, fmaxf(a[(q & 1) * 2], a[(q & 1) * 2 + 1]));
            }
        }
    mx = wave_max(mx);
    volatile __attribute__((address_space(3))) float* lmax = (volatile __attribute__((address_space(3))) float*)(lds + LDS_CTL + 64);
    lds_vu32* lcnt = (lds_vu32*)(lds + LDS_CTL + 128);
    if (lane == 0) lmax[wave] = mx;
    __syncthreads();
#pragma unroll
    for (int j = 0; j < 8; ++j) mx = fmaxf(mx, lmax[j]);
    const u32 qbase = OFF_Q + (u32)(tile * 2 + (wave >> 2)) * QPITCH;
    u32 L0, L1, mask = 0, pos;
    {
        const __amdgpu_buffer_rsrc_t rd = mk_rsrc(p.ws[wave], WS_USED);
        u32x2 gr; gr[0] = __float_as_uint(mx); gr[1] = seq;
        if (lane == 0) __builtin_amdgcn_raw_buffer_store_b64(gr, rd, AMAX + (rank * 128 + tile) * 8, 0, AUX_SYS);
        asm volatile("" ::: "memory");
        STAMP(3);
#pragma unroll
        for (int m = 0; m < 2; ++m)
#pragma unroll
            for (int j = 0; j < 4; ++j)
#pragma unroll
                for (int n = 0; n < 4; ++n) mask |= (acc[m][n][j] > 0.f ? 1u : 0u) << ((m * 4 + j) * 4 + n);
        u32 wtot;
        pos = wave_prefix(__builtin_popcount(mask), wtot);
        if (lane == 0) lcnt[wave] = wtot;
        __builtin_amdgcn_raw_buffer_store_b32(mask, rw, lane * 4, qbase + 16 + (wave & 3) * 256, AUX_SYS);
        asm volatile("s_waitcnt lgkmcnt(0)" ::: "memory");
        __builtin_amdgcn_s_barrier();
        asm volatile("" ::: "memory");
        u32 c[8], base = 0;
#pragma unroll
        for (int j = 0; j < 8; ++j) { c[j] = __builtin_amdgcn_readfirstlane(lcnt[j]); if ((j >> 2) == (wave >> 2) && j < wave) base += c[j]; }
        L0 = ((c[0] + c[1] + c[2] + c[3] + 7u) / 8u + 15u) & ~15u;
        L1 = ((c[4] + c[5] + c[6] + c[7] + 7u) / 8u + 15u) & ~15u;
        if (lane == 0) __builtin_amdgcn_raw_buffer_store_b32(base, rw, 0, qbase + (wave & 3) * 4, AUX_SYS);
        pos += base + 32768u + (u32)(wave >> 2) * 16384u;
        const u64 t0 = __builtin_amdgcn_s_memrealtime();
        u32x4 f;
        for (;;) {
            f = __builtin_amdgcn_raw_buffer_load_b128(rw, AMAX + wave * 1024 + lane * 16, 0, AUX_SYS);
            if (__all(f[1] == seq && f[3] == seq)) break;
            if (timed_out(p, seq, t0)) break;
            __builtin_amdgcn_s_sleep(1);
        }
        const u32 f0 = f[0], f2 = f[2];
        const float gm = wave_max(__uint_as_float(max(f0, f2)));
        if (lane == 0) lmax[8 + wave] = gm;
    }
    __syncthreads();
    STAMP(4);
    float amax = 0.f;
#pragma unroll
    for (int j = 0; j < 8; ++j) amax = fmaxf(amax, lmax[8 + j]);
    const bool bad = __hip_atomic_load((u32*)(wsl + TMO), __ATOMIC_RELAXED, __HIP_MEMORY_SCOPE_AGENT) == seq;
    const float inv = amax > 0.f ? 1.0f / (amax / 127.0f) : 0.f;
    {
        const int fr = lane & 15, fq = lane >> 4, wm = wave >> 1, wn = wave & 1;
#pragma unroll
        for (int m = 0; m < 2; ++m)
#pragma unroll
            for (int j = 0; j < 4; ++j) {
                u32 pk = 0;
#pragma unroll
                for (int n = 0; n < 4; ++n) {
                    const u32 q = (u32)(int)fminf(rintf(acc[m][n][j] * inv), 127.f);
                    const bool on = (mask >> ((m * 4 + j) * 4 + n)) & 1u;
                    *(lds + (on ? pos : 24576u + (u32)lane)) = (char)q;
                    pos += on;
                    pk |= q << (8 * n);
                }
                if (bad) pk = 0x80808080u;
                const int row = wm * 32 + m * 16 + fq * 4 + j;
                *(__attribute__((address_space(3))) u32*)(lds + row * 144 + fr * 8 + wn * 4) = pk;
            }
    }
    __syncthreads();
    if (wave < 2) {
        const u32 L = wave ? L1 : L0;
        const lds_char* pl = lds + 32768 + wave * 16384 + lane * 16;
        const u32 voff = (u32)lane * 16 < L ? (u32)lane * 16 : OOB;
        const u32x4 p7 = *(const __attribute__((address_space(3))) u32x4*)(pl + 7 * L);
#pragma unroll
        for (int j = 0; j < 7; ++j)
            __builtin_amdgcn_raw_buffer_store_b128((*(const __attribute__((address_space(3))) u32x4*)(pl + j * L) & 0x7f7f7f7fu) |
                                                   (((p7 >> j) & 0x01010101u) << 7), rw, voff,
                                                   OFF_Q + (u32)(tile * 2 + wave) * QPITCH + 1040u + j * L, AUX_SYS);
        asm volatile("s_waitcnt vmcnt(0)" ::: "memory");
    } else if (wave < 6) {
        const __amdgpu_buffer_rsrc_t ro = mk_rsrc(p.out[rank], OUT_BYTES);
        const int r0 = (wave - 2) * 32 + (lane >> 3), c0 = (lane & 7) * 16;
        const u32 voff = (u32)((grow(rank, 8 + rbk) + r0) * NDIM + gcol(rank, cb) + c0);
#pragma unroll
        for (int i = 0; i < 4; ++i)
            __builtin_amdgcn_raw_buffer_store_b128(*(const __attribute__((address_space(3))) u32x4*)(lds + (r0 + i * 8) * 144 + c0),
                                                   ro, voff, i * 8 * NDIM, AUX_SYS);
    }
    __builtin_amdgcn_s_barrier();
    STAMP(5);
    if (lane == 0) {
        const __amdgpu_buffer_rsrc_t rd = mk_rsrc(p.ws[wave], WS_USED);
        u32x2 gr; gr[0] = L0 | (L1 << 16); gr[1] = seq;
        __builtin_amdgcn_raw_buffer_store_b64(gr, rd, OFLAG + (rank * 128 + tile) * 8, 0, AUX_SYS);
        if (tile == 0 && wave == 0) __hip_atomic_store((u32*)(wsl + CNT), seq, __ATOMIC_RELAXED, __HIP_MEMORY_SCOPE_AGENT);
    }
}

extern "C" void kernel_launch(int rank, int nranks, ncclComm_t comm,
                              hipStream_t stream,
                              void* const* d_in, const int* in_sizes, int n_in,
                              void* d_out, int out_size,
                              const int* in_dtypes, int out_dtype,
                              void* const* d_out_all,
                              void* const* d_ws_all, size_t ws_bytes) {
    (void)comm; (void)in_dtypes; (void)out_dtype; (void)d_out;
    if (nranks != NR || (unsigned)rank >= (unsigned)NR || n_in < 2 || in_sizes[0] != MDIM * KSH ||
        in_sizes[1] != KSH * NDIM || out_size != MDIM * NDIM || ws_bytes < (size_t)OFF_DBG + (1u << 20)) {
        fprintf(stderr, "kernel_launch: unexpected problem shape\n");
        return;
    }
    static bool attr_set = false;
    if (!attr_set) {
        HIP_CHECK(hipFuncSetAttribute((const void*)fused_kernel, hipFuncAttributeMaxDynamicSharedMemorySize, LDS_BYTES));
        attr_set = true;
    }
    Params p;
    for (int r = 0; r < NR; ++r) { p.ws[r] = (char*)d_ws_all[r]; p.out[r] = (char*)d_out_all[r]; }
    p.x = (const char*)d_in[0];
    p.w = (const char*)d_in[1];
    p.rank = rank;
    hipLaunchKernelGGL(fused_kernel, dim3(GRID), dim3(THREADS), LDS_BYTES, stream, p);
    HIP_CHECK(hipGetLastError());
}
```

```cpp
#include <hip/hip_runtime.h>
#include <rccl/rccl.h>
#include <cstdio>
#include <cstdint>

#define HIP_CHECK(expr) do { hipError_t _e = (expr); if (_e != hipSuccess) { \
    fprintf(stderr, "HIP error %s at %s:%d\n", hipGetErrorString(_e), __FILE__, __LINE__); \
    return; } } while (0)

typedef unsigned int u32;
typedef unsigned long long u64;
typedef __attribute__((ext_vector_type(4))) u32 u32x4;
typedef __attribute__((ext_vector_type(2))) u32 u32x2;
typedef __attribute__((ext_vector_type(4))) float f32x4;
typedef __attribute__((ext_vector_type(8))) short bf16x8;
typedef __attribute__((address_space(3))) char lds_char;

namespace cfg {
constexpr int NR = 8;
constexpr int MDIM = 4096, NDIM = 4096, KSH = 512;
constexpr int NSTEP = 32;
constexpr int IMG = 16384;
constexpr int NBUF = 3;
constexpr int THREADS = 832;
constexpr int GRID = 256;
constexpr int NWV = 8;
#define N_PUSH 8
constexpr int NPUSH = N_PUSH;
#define O_DEPTH 2
constexpr int ODEPTH = O_DEPTH;
#define X_LATE0 2
#define FIRST_NEED 1u
#define N_IW 9
constexpr int NIW = N_IW;
constexpr int NU = 320;
constexpr int NO = 128;
constexpr u32 OFF_A = 0;
constexpr u32 OFF_B = OFF_A + NSTEP * 16 * IMG;
constexpr u32 EXP_X = OFF_B + NSTEP * 16 * IMG;
constexpr u32 EXP_W = EXP_X + 256 * IMG;
constexpr u32 OFF_P = EXP_W + 256 * IMG;
constexpr u32 OFF_P2 = OFF_P + 128 * 32768;
constexpr u32 OFF_Q = OFF_P2 + 128 * 32768;
constexpr u32 QPITCH = 7168;
constexpr u32 OFF_M = OFF_Q + 256 * QPITCH;
constexpr u32 OFF_CTL = OFF_M + 128 * 2048;
constexpr u32 FLAG_A = OFF_CTL;
constexpr u32 FLAG_B = FLAG_A + 8192;
constexpr u32 PFLAG = FLAG_B + 4096;
constexpr u32 PF2 = PFLAG + 4096;
constexpr u32 EFLAG = PF2 + 2048;
constexpr u32 AMAX = EFLAG + 16384;
constexpr u32 MFLAG = AMAX + 8192;
constexpr u32 OFLAG = MFLAG + 4096;
constexpr u32 CNT = OFLAG + 8192;
constexpr u32 TMO = CNT + 128;
constexpr u32 TICK = TMO + 128;
constexpr u32 WS_USED = TICK + 2048;
constexpr u32 OFF_DBG = 40u << 20;
constexpr u32 OUT_BYTES = (u32)MDIM * NDIM;
constexpr u32 IN_BYTES = (u32)MDIM * KSH * 2;
constexpr int LDS_P = NBUF * 2 * IMG;
constexpr int LDS_CTL = LDS_P + 32768;
constexpr int LDS_ORD = LDS_CTL + 256;
constexpr int LDS_BYTES = 151 * 1024;
constexpr u64 DEADLINE = 30000000ull;
constexpr int AUX_SYS = 17;
constexpr int AUX_SC1 = 16;
}
using namespace cfg;

struct Params {
    char* ws[NR];
    char* out[NR];
    const char* x;
    const char* w;
    char* wsl;
    int rank;
};

#define STAMP(idx) do { } while (0)

__device__ __forceinline__ __amdgpu_buffer_rsrc_t mk_rsrc(const void* base, u32 bytes) {
    return __builtin_amdgcn_make_buffer_rsrc((void*)base, (short)0, (int)bytes, 0x00020000);
}
__device__ __forceinline__ int row_quarter(int r, int lq) { int ri = (r >> 1) & 1; return (r >> 2) ? ri + 2 * lq : 2 * ri + lq; }
__device__ __forceinline__ int col_quarter(int r, int lq) { int rj = r & 1; return (r >> 2) ? rj + 2 * lq : 2 * rj + lq; }
__device__ __forceinline__ int grow(int r, int rb) {
    int cls = r >> 2, half = (rb >> 3) ? cls : 1 - cls;
    return row_quarter(r, (rb >> 2) & 1) * 1024 + half * 512 + (rb & 3) * 128;
}
__device__ __forceinline__ int gcol(int r, int cb) { return col_quarter(r, cb >> 3) * 1024 + (cb & 7) * 128; }

__device__ __forceinline__ void pair_of(int rank, int r8, int cb, int& partner, int& tile) {
    const int u = row_quarter(rank, r8 >> 2), v = col_quarter(rank, cb >> 3);
    int lqr, lqc;
    if (rank < 4) { partner = 4 + ((u & 1) << 1) + (v & 1); lqr = u >> 1; lqc = v >> 1; }
    else { partner = ((u >> 1) << 1) + (v >> 1); lqr = u & 1; lqc = v & 1; }
    tile = (lqr * 4 + (r8 & 3)) * 16 + lqc * 8 + (cb & 7);
}

__device__ __forceinline__ bool timed_out(const Params& p, u32 seq, u64 t0) {
    u32* tmo = (u32*)(p.wsl + TMO);
    if (__builtin_amdgcn_readfirstlane(__hip_atomic_load(tmo, __ATOMIC_RELAXED, __HIP_MEMORY_SCOPE_AGENT)) == seq ||
        __builtin_amdgcn_s_memrealtime() - t0 > DEADLINE) {
        __hip_atomic_store(tmo, seq, __ATOMIC_RELAXED, __HIP_MEMORY_SCOPE_AGENT);
        return true;
    }
    return false;
}

__device__ __forceinline__ float wave_max(float x) {
    u32 v = __float_as_uint(x);
    v = max(v, (u32)__builtin_amdgcn_update_dpp(0, (int)v, 0x111, 0xf, 0xf, true));
    v = max(v, (u32)__builtin_amdgcn_update_dpp(0, (int)v, 0x112, 0xf, 0xf, true));
    v = max(v, (u32)__builtin_amdgcn_update_dpp(0, (int)v, 0x114, 0xf, 0xf, true));
    v = max(v, (u32)__builtin_amdgcn_update_dpp(0, (int)v, 0x118, 0xf, 0xf, true));
    v = max(v, (u32)__builtin_amdgcn_update_dpp(0, (int)v, 0x142, 0xa, 0xf, true));
    v = max(v, (u32)__builtin_amdgcn_update_dpp(0, (int)v, 0x143, 0xc, 0xf, true));
    return __uint_as_float(__builtin_amdgcn_readlane(v, 63));
}
__device__ __forceinline__ void nibble_positions(u32 m16, u32& run, u32 (&pos)[4]) {
    const u32 c = __builtin_popcount(m16 & 15u) | (__builtin_popcount(m16 & 0xf0u) << 8) |
                  (__builtin_popcount(m16 & 0xf00u) << 16) | (__builtin_popcount(m16 & 0xf000u) << 24);
    u32 v = c;
    v += (u32)__builtin_amdgcn_update_dpp(0, (int)v, 0x111, 0xf, 0xf, true);
    v += (u32)__builtin_amdgcn_update_dpp(0, (int)v, 0x112, 0xf, 0xf, true);
    v += (u32)__builtin_amdgcn_update_dpp(0, (int)v, 0x114, 0xf, 0xf, true);
    v += (u32)__builtin_amdgcn_update_dpp(0, (int)v, 0x118, 0xf, 0xf, true);
    v += (u32)__builtin_amdgcn_update_dpp(0, (int)v, 0x142, 0xa, 0xf, true);
    v += (u32)__builtin_amdgcn_update_dpp(0, (int)v, 0x143, 0xc, 0xf, true);
    v -= c;
    const u32 e63 = __builtin_amdgcn_readlane(v, 63), c63 = __builtin_amdgcn_readlane(c, 63);
#pragma unroll
    for (int j = 0; j < 4; ++j) {
        pos[j] = run + ((v >> (8 * j)) & 255u);
        run += ((e63 >> (8 * j)) & 255u) + ((c63 >> (8 * j)) & 255u);
    }
}

struct Unit { u32 src; u32 dst; u32 flag; int isw; };

template <int NK>
__device__ __forceinline__ Unit decode(int u, int d, int sb, int sm) {
    constexpr int kbase = (NK == 5) ? 0 : 320;
    int kc, idx, g = 0, keep = 0;
    bool isw;
    if (u < 24 * NK) { kc = u / 24; idx = u % 24; isw = idx >= 16; idx &= 15; }
    else if (u < 48 * NK) {
        int v = u - 24 * NK, sg = v / (4 * NK);
        v %= 4 * NK; g = 1 + (sg >> 1); kc = v >> 2; idx = (v & 2) * 2 + (sg & 1) * 2 + (v & 1); isw = true;
    }
    else {
        const int v = u - 48 * NK, r = v >> 3, pre = NK == 5 ? X_LATE0 : 0, hw = r < pre || r >= pre + NK;
        kc = hw ? (r < pre ? r : r - NK) : r - pre; idx = hw * 8 + (v & 7); isw = false; keep = 8;
    }
    const int half = idx & 1, s = sb + sm * kc;
    Unit r;
    r.isw = isw;
    if (isw) {
        const int cb = (idx >> 2) * 8 + g * 2 + ((idx >> 1) & 1);
        r.src = (u32)(((kbase + kc * 64) * NDIM + gcol(d, cb) + half * 64) * 2);
        r.dst = OFF_B + (u32)((s * 16 + cb) * IMG + half * 128);
        r.flag = FLAG_B + (u32)((cb >> 1) * 512 + s * 16 + (cb & 1) * 8 + half * 4);
    } else {
        const int rb = (idx >> 1) + keep;
        r.src = (u32)(((grow(d, rb) + half * 64) * KSH + kbase + kc * 64) * 2);
        r.dst = OFF_A + (u32)((s * 16 + rb) * IMG + half * 1024);
        r.flag = FLAG_A + (u32)(rb * 512 + s * 16 + half * 4);
    }
    return r;
}

__device__ __forceinline__ u32x4 mk_desc(const void* base, u32 bytes) {
    const u64 a = (u64)base;
    u32x4 d;
    d[0] = __builtin_amdgcn_readfirstlane((u32)a);
    d[1] = __builtin_amdgcn_readfirstlane((u32)(a >> 32) & 0xffffu);
    d[2] = bytes;
    d[3] = 0x00020000u;
    return d;
}
#define LD16_LDS(ldsa, voff, desc, soff) do { u32 keep_m0; \
    asm volatile("s_mov_b32 %0, m0\n\ts_mov_b32 m0, %1\n\ts_nop 0\n\t" \
                 "buffer_load_dwordx4 %2, %3, %4 offen lds\n\ts_mov_b32 m0, %0" \
                 : "=&s"(keep_m0) : "s"(ldsa), "v"(voff), "s"(desc), "s"(soff) : "memory"); } while (0)
#define ST16_SYS(src, voff, desc, soff) \
    asm volatile("buffer_store_dwordx4 %0, %1, %2, %3 offen sc0 sc1\n\ts_nop 1" :: "v"(src), "v"(voff), "s"(desc), "s"(soff) : "memory")
#define ST4_SYS(src, voff, desc) \
    asm volatile("buffer_store_dword %0, %1, %2, 0 offen sc0 sc1\n\ts_nop 1" :: "v"(src), "v"(voff), "s"(desc) : "memory")
constexpr u32 OOB = 0x80000000u;

__device__ __forceinline__ void unit_load(u32 ldsa, const Params& p, const Unit& un, int lane) {
    const u32x4 rs = mk_desc(un.isw ? p.w : p.x, IN_BYTES);
    const u32 voff = (u32)(((lane >> 3) << (un.isw ? 16 : 10)) + (lane & 7) * 16);
    asm volatile("s_nop 4" ::: "memory");
#pragma unroll
    for (int i = 0; i < 8; ++i) LD16_LDS(ldsa + i * 1024, voff, rs, un.src + i * 8192);
}

__device__ __forceinline__ void unit_store(const lds_char* buf, u32x4 rd, const Unit& un, int lane) {
    const u32 voff = (u32)((lane >> 3) * 2048 + (lane & 7) * 16);
    const u32 ds = un.isw ? 256u : 128u;
    const bool isw = un.isw != 0;
    const int roff = isw ? lane * 16 : (lane & 7) * 128 + (lane >> 3) * 16;
    u32x4 v[8];
#pragma unroll
    for (int i = 0; i < 8; ++i) v[i] = *(const __attribute__((address_space(3))) u32x4*)(buf + roff + i * 1024);
#pragma unroll
    for (int t = 0; t < 8; ++t) {
        u32x4 o;
#pragma unroll
        for (int m = 0; m < 4; ++m) {
            const u32 tr = __builtin_amdgcn_perm(v[2 * m + 1][t >> 1], v[2 * m][t >> 1], (t & 1) ? 0x07060302u : 0x05040100u);
            o[m] = isw ? tr : v[t][m];
        }
        ST16_SYS(o, voff, rd, un.dst + t * ds);
    }
}

#define LD16_LDS_SYS(ldsa, voff, desc, soff, imm) do { u32 keep_m0; \
    asm volatile("s_mov_b32 %0, m0\n\ts_mov_b32 m0, %1\n\ts_nop 0\n\t" \
                 "buffer_load_dwordx4 %2, %3, %4 offen offset:" #imm " sc0 sc1 lds\n\ts_mov_b32 m0, %0" \
                 : "=&s"(keep_m0) : "s"(ldsa), "v"(voff), "s"(desc), "s"(soff) : "memory"); } while (0)

__device__ __forceinline__ void step_map(int s, int d, int& sb, int& sm) {
    if ((s >> 2) != (d >> 2)) { sb = 5 + ((s - d) & 3); sm = 4; }
    else { sb = 17 + ((s - d - 1) & 3); sm = 3; }
}

__device__ __forceinline__ void copy_unit(const Params& p, lds_char* lds, u32x4 rd, const Unit& un, int lane) {
    unit_load(__builtin_amdgcn_readfirstlane((u32)(uintptr_t)lds), p, un, lane);
    asm volatile("s_waitcnt vmcnt(0)" ::: "memory");
    unit_store(lds, rd, un, lane);
    asm volatile("s_waitcnt vmcnt(0)" ::: "memory");
}

__device__ __forceinline__ Unit export_unit_of(int e) {
    const int isw = e >> 9, h2 = (e >> 3) & 63, kc8 = e & 7;
    Unit un;
    un.isw = isw;
    un.src = isw ? (u32)((kc8 * 64 * NDIM + h2 * 64) * 2) : (u32)((h2 * 64 * KSH + kc8 * 64) * 2);
    un.dst = (isw ? EXP_W + (h2 & 1) * 128 : EXP_X + (h2 & 1) * 1024) + (u32)((kc8 * 32 + (h2 >> 1)) * IMG);
    un.flag = 0;
    return un;
}
__device__ __forceinline__ void export_unit(const Params& p, lds_char* lds, int e, u32 seq, int lane) {
    const int rank = p.rank, isw = e >> 9, h2 = (e >> 3) & 63, kc8 = e & 7;
    Unit un = export_unit_of(e);
    unit_store(lds, mk_desc(p.wsl, WS_USED), un, lane);
    asm volatile("s_waitcnt vmcnt(0)" ::: "memory");
    const int ccls = kc8 < 5 ? rank >> 2 : 1 - (rank >> 2), kcl = kc8 < 5 ? kc8 : kc8 - 5;
    const int q = h2 >> 4, sel = ccls ? q & 1 : q >> 1, lq = ccls ? q >> 1 : q & 1;
    const int b8 = (h2 >> 1) & 7, half = h2 & 1;
    const int blk = isw ? lq * 8 + b8 : ((b8 >> 2) == ccls ? 8 : 0) + lq * 4 + (b8 & 3);
    const u32 gflag = isw ? FLAG_B + (u32)((blk >> 1) * 512 + (blk & 1) * 8 + half * 4) : FLAG_A + (u32)(blk * 512 + half * 4);
#pragma unroll
    for (int o = 0; o < 2; ++o) {
        const int c = ccls * 4 + (isw ? o * 2 + sel : sel * 2 + o);
        const u32 fo = c == rank ? gflag + kcl * 16 : EFLAG + (u32)(rank * 2048 + (kcl * 64 + isw * 32 + lq * 16 + (h2 & 15)) * 4);
        const u32x4 rc = mk_desc(p.ws[c], WS_USED);
        asm volatile("s_nop 4" ::: "memory");
        ST4_SYS(seq, lane == 0 ? fo : OOB, rc);
    }
}

struct PUnit { u32 src; u32 dst; u32 flag; u32 wait; };
template <int NK>
__device__ __forceinline__ PUnit pdecode(int u, int rank, int d, int sb, int sm) {
    PUnit r;
    if (NK == 3 && u >= 64 * NK) {
        const int v = u - 64 * NK, g = v >> 5, tl = (v >> 2) & 7, q = v & 3, off = (tl & 3) * 16 + 2 * g + (tl >> 2);
        const int tile = ((d >> 1) & 1) * 64 + (d & 1) * 8 + off, ptile = ((rank >> 1) & 1) * 64 + (rank & 1) * 8 + off;
        r.src = OFF_P + (u32)(ptile * 32768 + q * 8192);
        r.dst = OFF_P2 + (u32)(tile * 32768 + q * 8192);
        r.flag = PF2 + (u32)(tile * 16 + q * 4);
        r.wait = (u32)(v >> 2) * 2;
        return r;
    }
    int kc, idx, g = 0, keep = 0;
    bool isw;
    if (u < 24 * NK) { kc = u / 24; idx = u % 24; isw = idx >= 16; idx &= 15; }
    else if (u < 48 * NK) {
        int v = u - 24 * NK, sg = v / (4 * NK);
        v %= 4 * NK; g = 1 + (sg >> 1); kc = v >> 2; idx = (v & 2) * 2 + (sg & 1) * 2 + (v & 1); isw = true;
    }
    else {
        const int v = u - 48 * NK, r = v >> 3, pre = NK == 5 ? X_LATE0 : 0, hw = r < pre || r >= pre + NK;
        kc = hw ? (r < pre ? r : r - NK) : r - pre; idx = hw * 8 + (v & 7); isw = false; keep = 8;
    }
    const int half = idx & 1, s = sb + sm * kc, kc8 = kc + (NK == 5 ? 0 : 5);
    r.wait = OOB;
    if (isw) {
        const int cb = (idx >> 2) * 8 + g * 2 + ((idx >> 1) & 1);
        r.src = EXP_W + (u32)((kc8 * 32 + (gcol(rank, cb) >> 7)) * IMG + half * 8192);
        r.dst = OFF_B + (u32)((s * 16 + cb) * IMG + half * 8192);
        r.flag = FLAG_B + (u32)((cb >> 1) * 512 + s * 16 + (cb & 1) * 8 + half * 4);
    } else {
        const int rb = (idx >> 1) + keep;
        r.src = EXP_X + (u32)((kc8 * 32 + (grow(rank, rb) >> 7)) * IMG + half * 8192);
        r.dst = OFF_A + (u32)((s * 16 + rb) * IMG + half * 8192);
        r.flag = FLAG_A + (u32)(rb * 512 + s * 16 + half * 4);
    }
    return r;
}

__device__ __forceinline__ void pull8k(u32 ldsa, u32x4 rs, u32 src, int lane) {
    const u32 voff = (u32)lane * 16;
    ldsa = __builtin_amdgcn_readfirstlane(ldsa);
    src = __builtin_amdgcn_readfirstlane(src);
    asm volatile("s_nop 4" :: "s"(ldsa), "s"(src) : "memory");
    LD16_LDS_SYS(ldsa, voff, rs, src, 0); LD16_LDS_SYS(ldsa, voff, rs, src, 1024);
    LD16_LDS_SYS(ldsa, voff, rs, src, 2048); LD16_LDS_SYS(ldsa, voff, rs, src, 3072);
    LD16_LDS_SYS(ldsa + 4096, voff, rs, src + 4096, 0); LD16_LDS_SYS(ldsa + 4096, voff, rs, src + 4096, 1024);
    LD16_LDS_SYS(ldsa + 4096, voff, rs, src + 4096, 2048); LD16_LDS_SYS(ldsa + 4096, voff, rs, src + 4096, 3072);
}

__device__ __forceinline__ void push_unit(const Params& p, lds_char* lds, int d, int u, u32 seq, int lane) {
    int sb, sm;
    step_map(p.rank, d, sb, sm);
    const Unit un = (d >> 2) == (p.rank >> 2) ? decode<5>(u, d, sb, sm) : decode<3>(u, d, sb, sm);
    const u32x4 rp = mk_desc(p.ws[d], WS_USED);
    copy_unit(p, lds, rp, un, lane);
    ST4_SYS(seq, lane == 0 ? un.flag : OOB, rp);
}

template <int NK>
__device__ __forceinline__ void mover(const Params& p, lds_char* lds, int d, int wv, u32 seq, int lane) {
    const int rank = p.rank;
    const __amdgpu_buffer_rsrc_t rw = mk_rsrc(p.wsl, WS_USED);
    const u32x4 rl = mk_desc(p.wsl, WS_USED), rp = mk_desc(p.ws[d], WS_USED);
    const u32 ldsa = __builtin_amdgcn_readfirstlane((u32)(uintptr_t)lds);
    int sb, sm;
    step_map(d, rank, sb, sm);
    {
        const u64 t0 = __builtin_amdgcn_s_memrealtime();
        for (int spins = 1;; ++spins) {
            bool ok = true;
#pragma unroll
            for (int i = 0; i < NK; ++i)
                ok &= __builtin_amdgcn_raw_buffer_load_b32(rw, EFLAG + (u32)(d * 2048 + i * 256 + lane * 4), 0, AUX_SYS) >= seq;
            if (__all(ok) || ((spins & 63) == 0 && timed_out(p, seq, t0))) break;
        }
    }
    const u32 pfo = PFLAG + (u32)((((d >> 1) & 1) * 64 + (d & 1) * 8 + ((lane >> 1) & 3) * 16 + 2 * (lane >> 4) + ((lane >> 3) & 1)) * 32 + (lane & 1) * 16);
    u64 pmask = 0;
    const u32 tbase = (seq - 1u) * (u32)(NU - NPUSH - NIW) - (u32)(NPUSH + 2 * NIW);
    int u = NPUSH + wv, u1 = NPUSH + NIW + wv;
    PUnit cur = pdecode<NK>(u, rank, d, sb, sm);
    u32 tk = (u32)__builtin_amdgcn_raw_ptr_buffer_atomic_add_i32(1, rw, lane == 0 ? TICK + d * 256 : OOB, 0, AUX_SC1);
    pull8k(ldsa, rp, cur.src, lane);
    u32x4 pf = {0u, 0u, 0u, 0u};
    u32 pflag = OOB;
    int par = 0;
    for (;;) {
        asm volatile("s_waitcnt vmcnt(0)" ::: "memory");
        const u32 n = __builtin_amdgcn_readfirstlane(tk) - tbase;
        if (NK == 3) pmask |= __ballot(pf[0] >= seq && pf[1] >= seq && pf[2] >= seq && pf[3] >= seq);
        asm volatile("" :: "s"(n), "s"(pmask) : "memory");
        const int u2 = (u1 < NU && n < (u32)NU) ? (int)n : NU;
        const PUnit nxt = pdecode<NK>(u1 < NU ? u1 : u, rank, d, sb, sm);
        const bool now = u1 < NU && !(NK == 3 && nxt.wait != OOB && ((pmask >> nxt.wait) & 3) != 3);
        if (now) {
            tk = (u32)__builtin_amdgcn_raw_ptr_buffer_atomic_add_i32(1, rw, (u2 < NU && lane == 0) ? TICK + d * 256 : OOB, 0, AUX_SC1);
            pull8k(ldsa + (par ^ 8192), rp, nxt.src, lane);
            if (NK == 3) pf = __builtin_amdgcn_raw_buffer_load_b128(rw, pfo, 0, AUX_SYS);
        }
        ST4_SYS(seq, pflag, rl);
        u32x4 v[8];
#pragma unroll
        for (int i = 0; i < 8; ++i) v[i] = *(const __attribute__((address_space(3))) u32x4*)(lds + par + lane * 16 + i * 1024);
#pragma unroll
        for (int i = 0; i < 8; ++i) ST16_SYS(v[i], (u32)lane * 16, rl, cur.dst + i * 1024);
        pflag = lane == 0 ? cur.flag : OOB;
        if (!now) {
            asm volatile("s_waitcnt vmcnt(0)" ::: "memory");
            ST4_SYS(seq, pflag, rl);
            pflag = OOB;
            if (u1 >= NU) break;
            const u64 t0 = __builtin_amdgcn_s_memrealtime();
            for (;;) {
                const u32x4 f = __builtin_amdgcn_raw_buffer_load_b128(rw, pfo, 0, AUX_SYS);
                pmask = __ballot(f[0] >= seq && f[1] >= seq && f[2] >= seq && f[3] >= seq);
                if (((pmask >> nxt.wait) & 3) == 3 || timed_out(p, seq, t0)) break;
                __builtin_amdgcn_s_sleep(1);
            }
            tk = (u32)__builtin_amdgcn_raw_ptr_buffer_atomic_add_i32(1, rw, (u2 < NU && lane == 0) ? TICK + d * 256 : OOB, 0, AUX_SC1);
            pull8k(ldsa + (par ^ 8192), rp, nxt.src, lane);
        }
        cur = nxt; u = u1; u1 = u2; par ^= 8192;
    }
}

constexpr int OB_NT = NO / NWV, OB_NB = 6;
constexpr int OB_IMG = 0, OB_MASK = 8 * 2304, OB_IN = OB_MASK + OB_NT * 2048, OB_PITCH = 8208,
              OB_LUT = OB_IN + 2 * OB_NB * OB_PITCH, OB_INFO = OB_LUT + 64, OB_LEN = OB_INFO + 64, OB_BASE = OB_LEN + 128,
              OB_END = OB_BASE + 1024;
typedef volatile __attribute__((address_space(3))) u32 lds_vu32;
#define MSTAMP(idx) do { } while (0)
__device__ __forceinline__ int tile_of(int wv, int i) { return i * 8 + ((wv + (i & 14)) & 7); }
__device__ __forceinline__ u32 plane_bytes(u32 n) { return ((n + 7u) / 8u + 15u) & ~15u; }
#define WG_BARRIER() do { asm volatile("s_waitcnt lgkmcnt(0)" ::: "memory"); __builtin_amdgcn_s_barrier(); asm volatile("" ::: "memory"); } while (0)

__device__ __forceinline__ void out_request(const Params& p, lds_char* lds, int d, int wv, u32 seq, int lane) {
    const int rank = p.rank;
    const __amdgpu_buffer_rsrc_t rw = mk_rsrc(p.wsl, WS_USED);
    const u32x4 rp = mk_desc(p.ws[d], WS_USED);
    const u32 ldsa = __builtin_amdgcn_readfirstlane((u32)(uintptr_t)lds);
    const u32 voff = (u32)lane * 16;
    u32 have = 0;
    u32 tm = 0, lens = 0;
    auto wait_streams = [&](u32 need) {
        const u64 t0 = __builtin_amdgcn_s_memrealtime();
        for (int spins = 1;; ++spins) {
            const u32x2 g = __builtin_amdgcn_raw_buffer_load_b64(rw, OFLAG + (u32)(d * 1024 + tile_of(wv, lane & 15) * 8), 0, AUX_SYS);
            lens = g[0];
            tm = __builtin_amdgcn_raw_buffer_load_b32(rw, TMO, 0, AUX_SC1);
            have = (u32)__ballot(g[1] == seq);
            if ((have & need) == need) break;
            if ((spins & 63) == 0 && timed_out(p, seq, t0)) { tm = seq; break; }
        }
    };
    for (u32 todo = 0xffffu; todo;) {
        if (!(have & todo)) {
            const u64 t0 = __builtin_amdgcn_s_memrealtime();
            for (int spins = 1; !(have & todo); ++spins) {
                have = (u32)__ballot(__builtin_amdgcn_raw_buffer_load_b32(rw, MFLAG + (u32)(d * 512 + tile_of(wv, lane & 15) * 4), 0, AUX_SYS) >= seq);
                if ((spins & 63) == 0 && timed_out(p, seq, t0)) have = 0xffffu;
            }
        }
        const int i = __builtin_ctz(have & todo);
        todo &= ~(1u << i);
        asm volatile("s_waitcnt vmcnt(6)" ::: "memory");
        const u32 src = __builtin_amdgcn_readfirstlane(OFF_M + (u32)tile_of(wv, i) * 2048);
        const u32 la = __builtin_amdgcn_readfirstlane(ldsa + OB_MASK + i * 2048);
        asm volatile("s_nop 4" :: "s"(la), "s"(src) : "memory");
        LD16_LDS_SYS(la, voff, rp, src, 0); LD16_LDS_SYS(la, voff, rp, src, 1024);
    }
    MSTAMP(3);
    constexpr int NH = 2 * OB_NT;
    int rbuf = 0;
    auto request = [&](int i) {
        const u32 bytes = i < NH ? 7u * min(((u32)__builtin_amdgcn_readlane(lens, i >> 1) >> ((i & 1) * 16)) & 0xffffu, 1024u) : 0u;
        const u32 src = __builtin_amdgcn_readfirstlane(OFF_Q + (u32)(tile_of(wv, (i >> 1) & 15) * 2 + (i & 1)) * QPITCH);
        const u32 la = __builtin_amdgcn_readfirstlane(ldsa + OB_IN + rbuf * OB_PITCH);
        rbuf = rbuf == 2 * OB_NB - 1 ? 0 : rbuf + 1;
        asm volatile("s_nop 4" :: "s"(la), "s"(src) : "memory");
        LD16_LDS_SYS(la, voff < bytes ? voff : OOB, rp, src, 0);
        LD16_LDS_SYS(la, 1024 + voff < bytes ? voff : OOB, rp, src, 1024);
        LD16_LDS_SYS(la, 2048 + voff < bytes ? voff : OOB, rp, src, 2048);
        LD16_LDS_SYS(la, 3072 + voff < bytes ? voff : OOB, rp, src, 3072);
        LD16_LDS_SYS(la + 4096, 4096 + voff < bytes ? voff : OOB, rp, src + 4096, 0);
        LD16_LDS_SYS(la + 4096, 5120 + voff < bytes ? voff : OOB, rp, src + 4096, 1024);
        LD16_LDS_SYS(la + 4096, 6144 + voff < bytes ? voff : OOB, rp, src + 4096, 2048);
    };
    wait_streams(FIRST_NEED);
    asm volatile("s_waitcnt vmcnt(0)" ::: "memory");
    MSTAMP(5);
    request(0); request(1);
    __builtin_amdgcn_s_barrier();
    if ((have & 0xffffu) != 0xffffu) wait_streams(0xffffu);
    MSTAMP(6);
    if (lane == 0) *(lds_vu32*)(lds + OB_INFO) = tm == seq;
    for (int i = 2; i < ODEPTH; ++i) request(i);
    for (int i = 0; i < NH; ++i) {
        asm volatile("s_waitcnt vmcnt(%0)" :: "n"(7 * (ODEPTH - 1)) : "memory");
        request(i + ODEPTH);
        if (i == 15) MSTAMP(7);
        if (i & 1) WG_BARRIER();
    }
    asm volatile("s_waitcnt vmcnt(0)" ::: "memory");
}

__device__ __forceinline__ void out_restore(lds_char* lds, int h, int lane) {
    WG_BARRIER();
    for (int i = 0, buf = h; i < OB_NT; ++i, buf = buf >= 2 * OB_NB - 2 ? h : buf + 2) {
        WG_BARRIER();
        const u32 L = __builtin_amdgcn_readfirstlane(*(lds_vu32*)(lds + OB_LEN + (2 * i + h) * 4));
        if ((u32)lane * 16 < L) {
            const lds_char* pl = lds + OB_IN + buf * OB_PITCH + lane * 16;
            u32x4 p7 = {0u, 0u, 0u, 0u};
#pragma unroll
            for (int j = 0; j < 7; ++j)
                p7 |= ((*(const __attribute__((address_space(3))) u32x4*)(pl + j * L) >> 7) & 0x01010101u) << j;
            *(__attribute__((address_space(3))) u32x4*)(pl + 7 * L) = p7;
        }
    }
    WG_BARRIER();
}

__device__ __forceinline__ void out_expand(const Params& p, lds_char* lds, int d, int wv, int e, int lane) {
    const int k = e & 3, h = e >> 2;
    const u32x4 ro = mk_desc(p.out[p.rank], OUT_BYTES);
    if (e == 0 && lane < 16) {
        u32 sel = 0, c = 0;
        for (int i = 0; i < 4; ++i) { sel |= (((lane >> i) & 1) ? c++ : 0x0cu) << (8 * i); }
        *(__attribute__((address_space(3))) u32*)(lds + OB_LUT + lane * 4) = sel;
    }
    WG_BARRIER();
#pragma unroll
    for (int q = 0; q < 4; ++q) {
        const int i = 4 * e + q;
        const u32x4 m = *(const __attribute__((address_space(3))) u32x4*)(lds + OB_MASK + i * 1024 + lane * 16);
        u32 c = 0;
#pragma unroll
        for (int x = 0; x < 4; ++x) c += __builtin_popcount(m[x] & 0xffffu) + (__builtin_popcount(m[x] >> 16) << 16);
        c += (u32)__builtin_amdgcn_update_dpp(0, (int)c, 0x111, 0xf, 0xf, true);
        c += (u32)__builtin_amdgcn_update_dpp(0, (int)c, 0x112, 0xf, 0xf, true);
        c += (u32)__builtin_amdgcn_update_dpp(0, (int)c, 0x114, 0xf, 0xf, true);
        c += (u32)__builtin_amdgcn_update_dpp(0, (int)c, 0x118, 0xf, 0xf, true);
        u32 bs[9];
        bs[0] = 0;
#pragma unroll
        for (int x = 0; x < 4; ++x) {
            const u32 t = __builtin_amdgcn_readlane(c, 16 * x + 15);
            bs[2 * x + 1] = bs[2 * x] + (t & 0xffffu);
            bs[2 * x + 2] = bs[2 * x + 1] + (t >> 16);
        }
        if (lane == 0) {
            *(__attribute__((address_space(3))) u32x4*)(lds + OB_BASE + i * 32) = u32x4{bs[0], bs[1], bs[2], bs[3]};
            *(__attribute__((address_space(3))) u32x4*)(lds + OB_BASE + i * 32 + 16) = u32x4{bs[4], bs[5], bs[6], bs[7]};
            *(__attribute__((address_space(3))) u32*)(lds + OB_LEN + i * 4) = plane_bytes(bs[8]);
        }
    }
    WG_BARRIER();
    lds_char* img = lds + OB_IMG + e * 2304;
    const int fr = lane & 15, fq = lane >> 4, sh = (k & 1) * 16;
    const u32 ovoff = (u32)((e * 16 + (lane >> 3)) * NDIM + (lane & 7) * 16);
    for (int i = 0, buf = h; i < OB_NT; ++i, buf = buf >= 2 * OB_NB - 2 ? h : buf + 2) {
        u32 pos[2][4], sel[2][4];
#pragma unroll
        for (int wn = 0; wn < 2; ++wn) {
            const int pw = (k >> 1) * 2 + wn;
            const u32 m = *(const __attribute__((address_space(3))) u32*)(lds + OB_MASK + i * 2048 + h * 1024 + pw * 256 + lane * 4);
            u32 run = *(const __attribute__((address_space(3))) u32*)(lds + OB_BASE + (i * 2 + h) * 32 + pw * 8 + (k & 1) * 4);
            nibble_positions(m >> sh, run, pos[wn]);
#pragma unroll
            for (int j = 0; j < 4; ++j)
                sel[wn][j] = *(const __attribute__((address_space(3))) u32*)(lds + OB_LUT + ((m >> (sh + 4 * j)) & 15u) * 4);
        }
        WG_BARRIER();
        const lds_char* in = lds + OB_IN + buf * OB_PITCH;
        const bool bad = *(lds_vu32*)(lds + OB_INFO) != 0;
#pragma unroll
        for (int j = 0; j < 4; ++j) {
            u32x2 o;
#pragma unroll
            for (int wn = 0; wn < 2; ++wn) {
                const __attribute__((address_space(3))) u32* sp = (const __attribute__((address_space(3))) u32*)(in + (pos[wn][j] & ~3u));
                o[wn] = __builtin_amdgcn_perm(0u, __builtin_amdgcn_alignbyte(sp[1], sp[0], pos[wn][j] & 3u), sel[wn][j]) & 0x7f7f7f7fu;
            }
            *(__attribute__((address_space(3))) u32x2*)(img + (fq * 4 + j) * 144 + fr * 8) = o;
        }
        const int tile = tile_of(wv, i);
        const u32 obase = (u32)(grow(d, 8 + (tile >> 4)) * NDIM + gcol(d, tile & 15));
#pragma unroll
        for (int r = 0; r < 2; ++r) {
            u32x4 v = *(const __attribute__((address_space(3))) u32x4*)(img + (r * 8 + (lane >> 3)) * 144 + (lane & 7) * 16);
            if (bad) v = u32x4{0x80808080u, 0x80808080u, 0x80808080u, 0x80808080u};
            ST16_SYS(v, ovoff, ro, obase + r * 8 * NDIM);
        }
    }
}

typedef volatile __attribute__((address_space(3))) int lds_int;
constexpr int SLOT = 2 * IMG;

__device__ __forceinline__ void glds4k(const char* g, lds_char* l) {
    const __attribute__((address_space(1))) void* gp = (const __attribute__((address_space(1))) void*)g;
    __attribute__((address_space(3))) void* lp = (__attribute__((address_space(3))) void*)l;
    __builtin_amdgcn_global_load_lds(gp, lp, 16, 0, 0);
    __builtin_amdgcn_global_load_lds(gp, lp, 16, 1024, 0);
    __builtin_amdgcn_global_load_lds(gp, lp, 16, 2048, 0);
    __builtin_amdgcn_global_load_lds(gp, lp, 16, 3072, 0);
}

__device__ __forceinline__ void gemm_poller(const Params& p, lds_char* lds, int rb, int cb, u32 seq, int lane) {
    const bool keep = rb >= 8;
    const __amdgpu_buffer_rsrc_t rw = mk_rsrc(p.wsl, WS_USED);
    const u32 foff = (lane < 32 ? FLAG_A + rb * 512 : FLAG_B + (cb >> 1) * 512 + (cb & 1) * 8) + (lane & 31) * 16;
    lds_int* lready = (lds_int*)(lds + LDS_CTL) + keep;
    lds_int* order = (lds_int*)(lds + LDS_ORD) + keep * 32;
    u32 listed = 0;
    const char* psrc = p.wsl + OFF_P2 + ((rb & 7) * 16 + cb) * 32768 + lane * 16;
    const u32 pfoff = PF2 + ((rb & 7) * 16 + cb) * 16 + (lane & 3) * 4;
    bool pdone = !keep;
    auto poll = [&]() -> u32 {
        const u32x2 f = __builtin_amdgcn_raw_buffer_load_b64(rw, foff, 0, AUX_SYS);
        const u64 m = __ballot(f[0] >= seq && f[1] >= seq);
        return (u32)m & (u32)(m >> 32);
    };
    u32 pfv = 0;
    auto pload = [&]() { pfv = __builtin_amdgcn_raw_buffer_load_b32(rw, pdone ? OOB : pfoff, 0, AUX_SYS); };
    auto pstep = [&]() {
        if (!pdone && __all(pfv >= seq)) {
#pragma unroll
            for (int i = 0; i < 8; ++i) glds4k(psrc + i * 4096, lds + LDS_P + i * 4096);
            pdone = true;
        }
    };
    int ready = 0;
    for (int t = 0; t < NSTEP; ++t) {
        if (ready < NSTEP && ready < t + NBUF) {
            const u64 t0 = __builtin_amdgcn_s_memrealtime();
            for (int spins = 1;; ++spins) {
                pload();
                u32 fresh = poll() & ~listed;
                if (!fresh && ready <= t && (spins & 63) == 0 && timed_out(p, seq, t0)) fresh = ~listed;
                if (fresh) {
                    if (lane < 32 && ((fresh >> lane) & 1u)) order[ready + __builtin_popcount(fresh & ((1u << lane) - 1u))] = lane;
                    listed |= fresh;
                    ready += __builtin_popcount(fresh);
                    if (lane == 0) lready[0] = ready;
                }
                if (ready > t + 1 || ready == NSTEP) break;
                if (ready > t && __builtin_amdgcn_readfirstlane(min(lready[2], lready[4])) > t) break;
                pstep();
            }
            asm volatile("s_waitcnt lgkmcnt(0)" ::: "memory");
        }
        __builtin_amdgcn_s_barrier();
    }
    if (!pdone) {
        const u64 t0 = __builtin_amdgcn_s_memrealtime();
        for (int spins = 1;; ++spins) {
            pload();
            pstep();
            if (pdone || ((spins & 63) == 0 && timed_out(p, seq, t0))) break;
        }
    }
    asm volatile("s_waitcnt vmcnt(0)" ::: "memory");
    __builtin_amdgcn_s_barrier();
}

__device__ __forceinline__ void gemm_loader(const Params& p, lds_char* lds, int rb, int cb, int lw, int lane) {
    const bool keep = rb >= 8;
    const int lp = lw >> 1, lh = lw & 1;
    lds_int* order = (lds_int*)(lds + LDS_ORD) + keep * 32;
    const char* wsl = p.wsl;
    lds_int* lready = (lds_int*)(lds + LDS_CTL) + keep;
    const char* ga = wsl + OFF_A + rb * IMG + lh * 8192 + lane * 16;
    const char* gb = wsl + OFF_B + cb * IMG + lh * 8192 + lane * 16;
    const char* ea = wsl + EXP_X + (grow(p.rank, rb) >> 7) * IMG + lh * 8192 + lane * 16;
    const char* eb = wsl + EXP_W + (gcol(p.rank, cb) >> 7) * IMG + lh * 8192 + lane * 16;
    int next = lp, ready = 0;
    auto try_issue = [&](int lim) {
        if (next >= lim || next >= NSTEP) return;
        if (ready <= next) ready = __builtin_amdgcn_readfirstlane(lready[0]);
        if (ready <= next) return;
        const int sl = __builtin_amdgcn_readfirstlane(order[next]) & 31;
        lds_char* dst = lds + (next % NBUF) * SLOT + lh * 8192;
        const bool own = sl < 5;
        const size_t so = (size_t)sl * (own ? 32 * IMG : 16 * IMG);
        const char* sa = (own ? ea : ga) + so;
        const char* sb = (own ? eb : gb) + so;
        glds4k(sa, dst); glds4k(sa + 4096, dst + 4096);
        glds4k(sb, dst + IMG); glds4k(sb + 4096, dst + IMG + 4096);
        next += 2;
    };
    for (int t = 0; t < NSTEP; ++t) {
        if ((t & 1) == lp) {
            while (next <= t) {
                try_issue(t + 1);
                if (next <= t) __builtin_amdgcn_s_sleep(1);
            }
            asm volatile("s_waitcnt vmcnt(0)" ::: "memory");
            if (lane == 0) lready[2 + 2 * lh] = t + 1;
        } else {
            while (next == t + 1 && next < NSTEP) {
                try_issue(t + 2);
                if (next == t + 1 && __builtin_amdgcn_readfirstlane(min(lready[2], lready[4])) > t) break;
            }
        }
        WG_BARRIER();
        try_issue(t + NBUF);
    }
    __builtin_amdgcn_s_barrier();
}

__device__ __forceinline__ void frag_mma(f32x4 (&acc)[2][4], const bf16x8 (&fa)[2], const bf16x8 (&fb)[4]) {
#pragma unroll
    for (int m = 0; m < 2; ++m)
#pragma unroll
        for (int n = 0; n < 4; ++n)
            acc[m][n] = __builtin_amdgcn_mfma_f32_16x16x32_bf16(fa[m], fb[n], acc[m][n], 0, 0, 0);
}
__device__ __forceinline__ void frag_read(bf16x8 (&fa)[2], bf16x8 (&fb)[4], const lds_char* a, const lds_char* b) {
#pragma unroll
    for (int m = 0; m < 2; ++m) fa[m] = *(const __attribute__((address_space(3))) bf16x8*)(a + m * 256);
#pragma unroll
    for (int n = 0; n < 4; ++n) fb[n] = *(const __attribute__((address_space(3))) bf16x8*)(b + n * 256);
}
__device__ __forceinline__ void interleave() {
#pragma unroll
    for (int i = 0; i < 6; ++i) {
        __builtin_amdgcn_sched_group_barrier(0x100, 1, 0);
        __builtin_amdgcn_sched_group_barrier(0x8, 1, 0);
    }
    __builtin_amdgcn_sched_barrier(0);
}

__device__ __forceinline__ void gemm_compute(lds_char* lds, int wave, int lane, f32x4 (&acc)[2][4]) {
    const int fr = lane & 15, fq = lane >> 4;
    const int aoff = fq * 2048 + ((wave >> 1) * 32 + fr) * 16;
    const int boff = IMG + fq * 2048 + (wave & 1) * 1024 + fr * 16;
    bf16x8 a0[2], b0[4], a1[2], b1[4];
#pragma unroll
    for (int m = 0; m < 2; ++m) {
#pragma unroll
        for (int n = 0; n < 4; ++n) acc[m][n] = f32x4{0.f, 0.f, 0.f, 0.f};
        a1[m] = bf16x8{0, 0, 0, 0, 0, 0, 0, 0};
    }
#pragma unroll
    for (int i = 0; i < 4; ++i) b1[i] = bf16x8{0, 0, 0, 0, 0, 0, 0, 0};
    for (int t = 0; t < NSTEP; ++t) {
        asm volatile("s_waitcnt lgkmcnt(0)" ::: "memory");
        __builtin_amdgcn_s_barrier();
        asm volatile("" ::: "memory");
        const lds_char* buf = lds + (t % NBUF) * SLOT;
        __builtin_amdgcn_sched_barrier(0);
        frag_read(a0, b0, buf + aoff, buf + boff);
        frag_mma(acc, a1, b1);
        interleave();
        frag_read(a1, b1, buf + aoff + 8192, buf + boff + 8192);
        frag_mma(acc, a0, b0);
        interleave();
    }
    frag_mma(acc, a1, b1);
    asm volatile("s_waitcnt lgkmcnt(0)" ::: "memory");
    __builtin_amdgcn_s_barrier();
    asm volatile("" ::: "memory");
}

__device__ __forceinline__ u32 pack_bf16(float lo, float hi) {
    typedef __attribute__((ext_vector_type(2))) __bf16 bf2;
    bf2 r; r[0] = (__bf16)lo; r[1] = (__bf16)hi;
    return __builtin_bit_cast(u32, r);
}

__device__ __forceinline__ u32x4 pack8(const f32x4& a, const f32x4& b) {
    u32x4 o;
    o[0] = pack_bf16(a[0], a[1]); o[1] = pack_bf16(a[2], a[3]);
    o[2] = pack_bf16(b[0], b[1]); o[3] = pack_bf16(b[2], b[3]);
    return o;
}

static_assert(OB_END <= LDS_BYTES && LDS_ORD + 256 <= LDS_BYTES, "LDS map");

__global__ void __launch_bounds__(THREADS, 4) fused_kernel(Params p) {
    extern __shared__ __attribute__((aligned(16))) char smem[];
    lds_char* lds = (lds_char*)smem;
    const int tid = threadIdx.x, lane = tid & 63;
    const int wave = __builtin_amdgcn_readfirstlane(tid >> 6);
    const int rank = p.rank;
    const int xcd = blockIdx.x & 7, slot = blockIdx.x >> 3;
    char* wsl = p.wsl;
    const u32 cntv = __hip_atomic_load((u32*)(wsl + CNT), __ATOMIC_RELAXED, __HIP_MEMORY_SCOPE_AGENT);
    const int rbk = (xcd & 1) * 4 + (slot & 3);
    const int cb = (xcd >> 1) * 4 + ((slot >> 2) & 3);
    const int tile = rbk * 16 + cb;
    if (slot >= 16 && wave < 8)
        unit_load(__builtin_amdgcn_readfirstlane((u32)(uintptr_t)(lds + 4 * IMG + wave * 8192)), p, export_unit_of(wave * 128 + tile), lane);
    asm volatile("s_waitcnt vmcnt(0)" ::: "memory");
    const u32 seq = __builtin_amdgcn_readfirstlane(cntv) + 1u;
    STAMP(0);

    if (slot < 8) {
        const int d = (rank + slot) & 7, wv = (xcd + slot) & 7;
        if (slot == 0) {
            if (NIW > 8 && wave == 0 && xcd) {
                const int d9 = (rank + xcd) & 7;
                if ((d9 >> 2) == (rank >> 2)) mover<5>(p, lds, d9, 8, seq, lane);
                else mover<3>(p, lds, d9, 8, seq, lane);
            }
            return;
        }
        if (wave > 10) return;
        if (wave > 8) { out_restore(lds, wave - 9, lane); return; }
        if (wave) { out_expand(p, lds, d, wv, wave - 1, lane); return; }
        if ((d >> 2) == (rank >> 2)) mover<5>(p, lds, d, wv, seq, lane);
        else mover<3>(p, lds, d, wv, seq, lane);
        out_request(p, lds, d, wv, seq, lane);
        return;
    }

    if (wave == 8) {
        if (lane < 6) ((lds_int*)(lds + LDS_CTL))[lane] = 0;
    }
    WG_BARRIER();
    const int srb = (xcd & 1) * 4 + (slot & 3);
    const int scb = (xcd >> 2) * 8 + (slot < 16 ? 4 : 0) + ((slot >> 2) & 1) * 2 + ((xcd >> 1) & 1);
    const int first = slot < 24 ? 0 : 1, last = slot < 16 ? 0 : 1;
    if (wave >= 8) {
        for (int pass = first; pass <= last; ++pass) {
            const int rb = pass ? 8 + rbk : srb, c = pass ? cb : scb;
            if (wave == 12) gemm_poller(p, lds, rb, c, seq, lane);
            else gemm_loader(p, lds, rb, c, wave - 8, lane);
        }
        return;
    }
    if (slot >= 16) export_unit(p, lds + 4 * IMG + wave * 8192, wave * 128 + tile, seq, lane);
    else if (wave * 64 + (slot - 8) * 8 + xcd < 7 * NPUSH) {
        const int id = wave * 64 + (slot - 8) * 8 + xcd;
        push_unit(p, lds + 4 * IMG + wave * 8192, (rank + 1 + id % 7) & 7, id / 7, seq, lane);
    }
    STAMP(6);
    const __amdgpu_buffer_rsrc_t rw = mk_rsrc(wsl, WS_USED);
    f32x4 acc[2][4];
    for (int pass = first;; ++pass) {
        gemm_compute(lds, wave, lane, acc);
        if (pass) break;
        STAMP(7);
        int partner, ptile;
        pair_of(rank, srb, scb, partner, ptile);
        const __amdgpu_buffer_rsrc_t rd = mk_rsrc(p.ws[partner], WS_USED);
#pragma unroll
        for (int m = 0; m < 2; ++m)
#pragma unroll
            for (int h = 0; h < 2; ++h)
                __builtin_amdgcn_raw_buffer_store_b128(pack8(acc[m][2 * h], acc[m][2 * h + 1]), rw, tid * 16,
                                                       OFF_P + (srb * 16 + scb) * 32768 + (m * 2 + h) * 8192, AUX_SYS);
        asm volatile("s_waitcnt vmcnt(0)" ::: "memory");
        if (lane == 0) __builtin_amdgcn_raw_buffer_store_b32(seq, rd, PFLAG + ptile * 32 + wave * 4, 0, AUX_SYS);
        STAMP(8);
        if (!last) return;
    }
    STAMP(2);
    float mx = 0.f;
#pragma unroll
    for (int m = 0; m < 2; ++m)
#pragma unroll
        for (int h = 0; h < 2; ++h) {
            const u32x4 o = *(const __attribute__((address_space(3))) u32x4*)(lds + LDS_P + (m * 2 + h) * 8192 + tid * 16);
#pragma unroll
            for (int q = 0; q < 4; ++q) {
                float lo = __builtin_bit_cast(float, o[q] << 16), hi = __builtin_bit_cast(float, o[q] & 0xffff0000u);
                f32x4& a = acc[m][2 * h + (q >> 1)];
                a[(q & 1) * 2] = fmaxf(a[(q & 1) * 2] + lo, 0.f);
                a[(q & 1) * 2 + 1] = fmaxf(a[(q & 1) * 2 + 1] + hi, 0.f);
                mx = fmaxf(mx, fmaxf(a[(q & 1) * 2], a[(q & 1) * 2 + 1]));
            }
        }
    mx = wave_max(mx);
    volatile __attribute__((address_space(3))) float* lmax = (volatile __attribute__((address_space(3))) float*)(lds + LDS_CTL + 64);
    lds_vu32* lcnt = (lds_vu32*)(lds + LDS_CTL + 128);
    if (lane == 0) lmax[wave] = mx;
    __syncthreads();
#pragma unroll
    for (int j = 0; j < 8; ++j) mx = fmaxf(mx, lmax[j]);
    u32 L, L01, mask = 0;
    f32x4 col[8];
    {
        const __amdgpu_buffer_rsrc_t rd = mk_rsrc(p.ws[wave], WS_USED);
        u32x2 gr; gr[0] = __float_as_uint(mx); gr[1] = seq;
        if (lane == 0) __builtin_amdgcn_raw_buffer_store_b64(gr, rd, AMAX + (rank * 128 + tile) * 8, 0, AUX_SYS);
        asm volatile("" ::: "memory");
        STAMP(3);
#pragma unroll
        for (int m = 0; m < 2; ++m)
#pragma unroll
            for (int j = 0; j < 4; ++j)
#pragma unroll
                for (int n = 0; n < 4; ++n) mask |= (acc[m][n][j] > 0.f ? 1u : 0u) << ((m * 4 + j) * 4 + n);
        __builtin_amdgcn_raw_buffer_store_b32(mask, rw, tid * 4, OFF_M + tile * 2048, AUX_SYS);
        u32 pos[2][4], wtot = 0;
        nibble_positions(mask, wtot, pos[0]);
        nibble_positions(mask >> 16, wtot, pos[1]);
        if (lane == 0) lcnt[wave] = wtot;
        asm volatile("s_waitcnt vmcnt(0)" ::: "memory");
        WG_BARRIER();
        if (lane == 0) __builtin_amdgcn_raw_buffer_store_b32(seq, rd, MFLAG + (rank * 128 + tile) * 4, 0, AUX_SYS);
        u32 base = 0, n[2] = {0u, 0u};
#pragma unroll
        for (int j = 0; j < 8; ++j) {
            const u32 c = __builtin_amdgcn_readfirstlane(lcnt[j]);
            if ((j >> 2) == (wave >> 2) && j < wave) base += c;
            n[j >> 2] += c;
        }
        L01 = plane_bytes(n[0]) | (plane_bytes(n[1]) << 16);
        L = (wave >> 2) ? L01 >> 16 : L01 & 0xffffu;
        lds_char* fs = lds + 32768 + (wave >> 2) * 32768;
#pragma unroll
        for (int dw = 0; dw < 8; ++dw) {
            u32 ps = (base + pos[dw >> 2][dw & 3]) * 4;
#pragma unroll
            for (int c = 0; c < 4; ++c) {
                const bool on = (mask >> (4 * dw + c)) & 1u;
                *(__attribute__((address_space(3))) float*)(on ? fs + ps : lds + 24576 + lane * 4) = acc[dw >> 2][c][dw & 3];
                ps += on ? 4u : 0u;
            }
        }
        WG_BARRIER();
#pragma unroll
        for (int j = 0; j < 8; ++j)
            col[j] = *(const __attribute__((address_space(3))) f32x4*)(fs + ((j * L + ((tid & 255) * 4 < L ? (tid & 255) * 4 : 0)) * 4));
        const u64 t0 = __builtin_amdgcn_s_memrealtime();
        u32x4 f;
        u32 tm;
        for (int spins = 1;; ++spins) {
            f = __builtin_amdgcn_raw_buffer_load_b128(rw, AMAX + wave * 1024 + lane * 16, 0, AUX_SYS);
            tm = __builtin_amdgcn_raw_buffer_load_b32(rw, TMO, 0, AUX_SC1);
            if (__all(f[1] == seq && f[3] == seq)) break;
            if ((spins & 63) == 0 && timed_out(p, seq, t0)) { tm = seq; break; }
        }
        const u32 f0 = f[0], f2 = f[2];
        const float gm = wave_max(__uint_as_float(max(f0, f2)));
        if (lane == 0) { lmax[8 + wave] = gm; lcnt[8 + wave] = tm; }
    }
    __syncthreads();
    STAMP(4);
    float amax = 0.f;
#pragma unroll
    for (int j = 0; j < 8; ++j) amax = fmaxf(amax, lmax[8 + j]);
    bool bad = false;
#pragma unroll
    for (int j = 0; j < 8; ++j) bad = bad || lcnt[8 + j] == seq;
    const float inv = amax > 0.f ? 1.0f / (amax / 127.0f) : 0.f;
    {
        u32 q[8];
#pragma unroll
        for (int j = 0; j < 8; ++j) {
            q[j] = 0;
#pragma unroll
            for (int c = 0; c < 4; ++c) q[j] = __builtin_amdgcn_cvt_pk_u8_f32(rintf(col[j][c] * inv), c, q[j]);
        }
        const u32 voff = (u32)(tid & 255) * 4 < L ? (u32)(tid & 255) * 4 : OOB;
#pragma unroll
        for (int j = 0; j < 7; ++j)
            __builtin_amdgcn_raw_buffer_store_b32((q[j] & 0x7f7f7f7fu) | (((q[7] >> j) & 0x01010101u) << 7), rw, voff,
                                                  OFF_Q + (u32)(tile * 2 + (wave >> 2)) * QPITCH + j * L, AUX_SYS);
        STAMP(11);
    }
    {
        const int fr = lane & 15, fq = lane >> 4, wm = wave >> 1, wn = wave & 1;
#pragma unroll
        for (int m = 0; m < 2; ++m)
#pragma unroll
            for (int j = 0; j < 4; ++j) {
                u32 pk = 0;
#pragma unroll
                for (int n = 0; n < 4; ++n) pk = __builtin_amdgcn_cvt_pk_u8_f32(rintf(acc[m][n][j] * inv), n, pk);
                if (bad) pk = 0x80808080u;
                const int row = wm * 32 + m * 16 + fq * 4 + j;
                *(__attribute__((address_space(3))) u32*)(lds + row * 144 + fr * 8 + wn * 4) = pk;
            }
    }
    asm volatile("s_waitcnt vmcnt(0)" ::: "memory");
    WG_BARRIER();
    STAMP(5);
    if (lane == 0) {
        const __amdgpu_buffer_rsrc_t rd = mk_rsrc(p.ws[wave], WS_USED);
        __builtin_amdgcn_raw_buffer_store_b64(u32x2{L01, seq}, rd, OFLAG + (rank * 128 + tile) * 8, 0, AUX_SYS);
        if (tile == 0 && wave == 0) __hip_atomic_store((u32*)(wsl + CNT), seq, __ATOMIC_RELAXED, __HIP_MEMORY_SCOPE_AGENT);
    }
    const __amdgpu_buffer_rsrc_t ro = mk_rsrc(p.out[rank], OUT_BYTES);
    const int r0 = wave * 16 + (lane >> 3), c0 = (lane & 7) * 16;
    const u32 voff = (u32)((grow(rank, 8 + rbk) + r0) * NDIM + gcol(rank, cb) + c0);
#pragma unroll
    for (int i = 0; i < 2; ++i)
        __builtin_amdgcn_raw_buffer_store_b128(*(const __attribute__((address_space(3))) u32x4*)(lds + (r0 + i * 8) * 144 + c0),
                                               ro, voff, i * 8 * NDIM, AUX_SYS);
}

extern "C" void kernel_launch(int rank, int nranks, ncclComm_t comm,
                              hipStream_t stream,
                              void* const* d_in, const int* in_sizes, int n_in,
                              void* d_out, int out_size,
                              const int* in_dtypes, int out_dtype,
                              void* const* d_out_all,
                              void* const* d_ws_all, size_t ws_bytes) {
    (void)comm; (void)in_dtypes; (void)out_dtype; (void)d_out;
    if (nranks != NR || (unsigned)rank >= (unsigned)NR || n_in < 2 || in_sizes[0] != MDIM * KSH ||
        in_sizes[1] != KSH * NDIM || out_size != MDIM * NDIM || ws_bytes < (size_t)OFF_DBG + (1u << 20)) {
        fprintf(stderr, "kernel_launch: unexpected problem shape\n");
        return;
    }
    static bool attr_set = false;
    if (!attr_set) {
        HIP_CHECK(hipFuncSetAttribute((const void*)fused_kernel, hipFuncAttributeMaxDynamicSharedMemorySize, LDS_BYTES));
        attr_set = true;
    }
    Params p;
    for (int r = 0; r < NR; ++r) { p.ws[r] = (char*)d_ws_all[r]; p.out[r] = (char*)d_out_all[r]; }
    p.x = (const char*)d_in[0];
    p.w = (const char*)d_in[1];
    p.rank = rank;
    p.wsl = (char*)d_ws_all[rank];
    hipLaunchKernelGGL(fused_kernel, dim3(GRID), dim3(THREADS), LDS_BYTES, stream, p);
    HIP_CHECK(hipGetLastError());
}
```

```cpp
#include <hip/hip_runtime.h>
#include <rccl/rccl.h>
#include <cstdio>
#include <cstdint>

#define HIP_CHECK(expr) do { hipError_t _e = (expr); if (_e != hipSuccess) { \
    fprintf(stderr, "HIP error %s at %s:%d\n", hipGetErrorString(_e), __FILE__, __LINE__); \
    return; } } while (0)

typedef unsigned int u32;
typedef unsigned long long u64;
typedef __attribute__((ext_vector_type(4))) u32 u32x4;
typedef __attribute__((ext_vector_type(2))) u32 u32x2;
typedef __attribute__((ext_vector_type(4))) float f32x4;
typedef __attribute__((ext_vector_type(8))) short bf16x8;
typedef __attribute__((address_space(3))) char lds_char;

namespace cfg {
constexpr int NR = 8;
constexpr int MDIM = 4096, NDIM = 4096, KSH = 512;
constexpr int NSTEP = 32;
constexpr int IMG = 16384;
constexpr int NBUF = 3;
constexpr int THREADS = 832;
constexpr int GRID = 256;
constexpr int NWV = 8;
constexpr int NIW = 9;
constexpr int NPUSH1 = 8, NPUSH = 16;
constexpr u64 PUSH_DELAY = 320;
constexpr int ODEPTH = 2;
constexpr int X_LATE0 = 2;
constexpr int NU = 320;
constexpr int NO = 128;
constexpr u32 OFF_A = 0;
constexpr u32 OFF_B = OFF_A + NSTEP * 16 * IMG;
constexpr u32 EXP_X = OFF_B + NSTEP * 16 * IMG;
constexpr u32 EXP_W = EXP_X + 256 * IMG;
constexpr u32 OFF_P = EXP_W + 256 * IMG;
constexpr u32 OFF_P2 = OFF_P + 128 * 32768;
constexpr u32 OFF_Q = OFF_P2 + 128 * 32768;
constexpr u32 QPITCH = 7168;
constexpr u32 OFF_M = OFF_Q + 256 * QPITCH;
constexpr u32 OFF_CTL = OFF_M + 128 * 2048;
constexpr u32 FLAG_A = OFF_CTL;
constexpr u32 FLAG_B = FLAG_A + 8192;
constexpr u32 PFLAG = FLAG_B + 4096;
constexpr u32 PF2 = PFLAG + 4096;
constexpr u32 EFLAG = PF2 + 2048;
constexpr u32 AMAX = EFLAG + 16384;
constexpr u32 MFLAG = AMAX + 8192;
constexpr u32 OFLAG = MFLAG + 4096;
constexpr u32 CNT = OFLAG + 8192;
constexpr u32 TMO = CNT + 128;
constexpr u32 TICK = TMO + 128;
constexpr u32 WS_USED = TICK + 2048;
constexpr u32 OUT_BYTES = (u32)MDIM * NDIM;
constexpr u32 IN_BYTES = (u32)MDIM * KSH * 2;
constexpr int LDS_P = NBUF * 2 * IMG;
constexpr int LDS_CTL = LDS_P + 32768;
constexpr int LDS_ORD = LDS_CTL + 256;
constexpr int LDS_BYTES = 151 * 1024;
constexpr u64 DEADLINE = 30000000ull;
constexpr int AUX_SYS = 17;
constexpr int AUX_SC1 = 16;
}
using namespace cfg;

struct Params {
    char* ws[NR];
    char* out[NR];
    const char* x;
    const char* w;
    char* wsl;
    int rank;
};

__device__ __forceinline__ __amdgpu_buffer_rsrc_t mk_rsrc(const void* base, u32 bytes) {
    return __builtin_amdgcn_make_buffer_rsrc((void*)base, (short)0, (int)bytes, 0x00020000);
}
__device__ __forceinline__ int row_quarter(int r, int lq) { int ri = (r >> 1) & 1; return (r >> 2) ? ri + 2 * lq : 2 * ri + lq; }
__device__ __forceinline__ int col_quarter(int r, int lq) { int rj = r & 1; return (r >> 2) ? rj + 2 * lq : 2 * rj + lq; }
__device__ __forceinline__ int grow(int r, int rb) {
    int cls = r >> 2, half = (rb >> 3) ? cls : 1 - cls;
    return row_quarter(r, (rb >> 2) & 1) * 1024 + half * 512 + (rb & 3) * 128;
}
__device__ __forceinline__ int gcol(int r, int cb) { return col_quarter(r, cb >> 3) * 1024 + (cb & 7) * 128; }

__device__ __forceinline__ void pair_of(int rank, int r8, int cb, int& partner, int& tile) {
    const int u = row_quarter(rank, r8 >> 2), v = col_quarter(rank, cb >> 3);
    int lqr, lqc;
    if (rank < 4) { partner = 4 + ((u & 1) << 1) + (v & 1); lqr = u >> 1; lqc = v >> 1; }
    else { partner = ((u >> 1) << 1) + (v >> 1); lqr = u & 1; lqc = v & 1; }
    tile = (lqr * 4 + (r8 & 3)) * 16 + lqc * 8 + (cb & 7);
}

__device__ __forceinline__ bool timed_out(const Params& p, u32 seq, u64 t0) {
    u32* tmo = (u32*)(p.wsl + TMO);
    if (__builtin_amdgcn_readfirstlane(__hip_atomic_load(tmo, __ATOMIC_RELAXED, __HIP_MEMORY_SCOPE_AGENT)) == seq ||
        __builtin_amdgcn_s_memrealtime() - t0 > DEADLINE) {
        __hip_atomic_store(tmo, seq, __ATOMIC_RELAXED, __HIP_MEMORY_SCOPE_AGENT);
        return true;
    }
    return false;
}

__device__ __forceinline__ float wave_max(float x) {
    u32 v = __float_as_uint(x);
    v = max(v, (u32)__builtin_amdgcn_update_dpp(0, (int)v, 0x111, 0xf, 0xf, true));
    v = max(v, (u32)__builtin_amdgcn_update_dpp(0, (int)v, 0x112, 0xf, 0xf, true));
    v = max(v, (u32)__builtin_amdgcn_update_dpp(0, (int)v, 0x114, 0xf, 0xf, true));
    v = max(v, (u32)__builtin_amdgcn_update_dpp(0, (int)v, 0x118, 0xf, 0xf, true));
    v = max(v, (u32)__builtin_amdgcn_update_dpp(0, (int)v, 0x142, 0xa, 0xf, true));
    v = max(v, (u32)__builtin_amdgcn_update_dpp(0, (int)v, 0x143, 0xc, 0xf, true));
    return __uint_as_float(__builtin_amdgcn_readlane(v, 63));
}
__device__ __forceinline__ void nibble_positions(u32 m16, u32& run, u32 (&pos)[4]) {
    const u32 c = __builtin_popcount(m16 & 15u) | (__builtin_popcount(m16 & 0xf0u) << 8) |
                  (__builtin_popcount(m16 & 0xf00u) << 16) | (__builtin_popcount(m16 & 0xf000u) << 24);
    u32 v = c;
    v += (u32)__builtin_amdgcn_update_dpp(0, (int)v, 0x111, 0xf, 0xf, true);
    v += (u32)__builtin_amdgcn_update_dpp(0, (int)v, 0x112, 0xf, 0xf, true);
    v += (u32)__builtin_amdgcn_update_dpp(0, (int)v, 0x114, 0xf, 0xf, true);
    v += (u32)__builtin_amdgcn_update_dpp(0, (int)v, 0x118, 0xf, 0xf, true);
    v += (u32)__builtin_amdgcn_update_dpp(0, (int)v, 0x142, 0xa, 0xf, true);
    v += (u32)__builtin_amdgcn_update_dpp(0, (int)v, 0x143, 0xc, 0xf, true);
    v -= c;
    const u32 e63 = __builtin_amdgcn_readlane(v, 63), c63 = __builtin_amdgcn_readlane(c, 63);
#pragma unroll
    for (int j = 0; j < 4; ++j) {
        pos[j] = run + ((v >> (8 * j)) & 255u);
        run += ((e63 >> (8 * j)) & 255u) + ((c63 >> (8 * j)) & 255u);
    }
}

struct Unit { u32 src; u32 dst; u32 flag; int isw; };

template <int NK>
__device__ __forceinline__ Unit decode(int u, int d, int sb, int sm) {
    constexpr int kbase = (NK == 5) ? 0 : 320;
    int kc, idx, g = 0, keep = 0;
    bool isw;
    if (u < 24 * NK) { kc = u / 24; idx = u % 24; isw = idx >= 16; idx &= 15; }
    else if (u < 48 * NK) {
        int v = u - 24 * NK, sg = v / (4 * NK);
        v %= 4 * NK; g = 1 + (sg >> 1); kc = v >> 2; idx = (v & 2) * 2 + (sg & 1) * 2 + (v & 1); isw = true;
    }
    else {
        const int v = u - 48 * NK, r = v >> 3, pre = NK == 5 ? X_LATE0 : 0, hw = r < pre || r >= pre + NK;
        kc = hw ? (r < pre ? r : r - NK) : r - pre; idx = hw * 8 + (v & 7); isw = false; keep = 8;
    }
    const int half = idx & 1, s = sb + sm * kc;
    Unit r;
    r.isw = isw;
    if (isw) {
        const int cb = (idx >> 2) * 8 + g * 2 + ((idx >> 1) & 1);
        r.src = (u32)(((kbase + kc * 64) * NDIM + gcol(d, cb) + half * 64) * 2);
        r.dst = OFF_B + (u32)((s * 16 + cb) * IMG + half * 128);
        r.flag = FLAG_B + (u32)((cb >> 1) * 512 + s * 16 + (cb & 1) * 8 + half * 4);
    } else {
        const int rb = (idx >> 1) + keep;
        r.src = (u32)(((grow(d, rb) + half * 64) * KSH + kbase + kc * 64) * 2);
        r.dst = OFF_A + (u32)((s * 16 + rb) * IMG + half * 1024);
        r.flag = FLAG_A + (u32)(rb * 512 + s * 16 + half * 4);
    }
    return r;
}

__device__ __forceinline__ u32x4 mk_desc(const void* base, u32 bytes) {
    const u64 a = (u64)base;
    u32x4 d;
    d[0] = __builtin_amdgcn_readfirstlane((u32)a);
    d[1] = __builtin_amdgcn_readfirstlane((u32)(a >> 32) & 0xffffu);
    d[2] = bytes;
    d[3] = 0x00020000u;
    return d;
}
#define LD16_LDS(ldsa, voff, desc, soff) do { u32 keep_m0; \
    asm volatile("s_mov_b32 %0, m0\n\ts_mov_b32 m0, %1\n\ts_nop 0\n\t" \
                 "buffer_load_dwordx4 %2, %3, %4 offen lds\n\ts_mov_b32 m0, %0" \
                 : "=&s"(keep_m0) : "s"(ldsa), "v"(voff), "s"(desc), "s"(soff) : "memory"); } while (0)
#define ST16_SYS(src, voff, desc, soff) \
    asm volatile("buffer_store_dwordx4 %0, %1, %2, %3 offen sc0 sc1\n\ts_nop 1" :: "v"(src), "v"(voff), "s"(desc), "s"(soff) : "memory")
#define ST4_SYS(src, voff, desc) \
    asm volatile("buffer_store_dword %0, %1, %2, 0 offen sc0 sc1\n\ts_nop 1" :: "v"(src), "v"(voff), "s"(desc) : "memory")
constexpr u32 OOB = 0x80000000u;

__device__ __forceinline__ void unit_load(u32 ldsa, const Params& p, const Unit& un, int lane) {
    const u32x4 rs = mk_desc(un.isw ? p.w : p.x, IN_BYTES);
    const u32 voff = (u32)(((lane >> 3) << (un.isw ? 16 : 10)) + (lane & 7) * 16);
    asm volatile("s_nop 4" :: "s"(rs), "s"(un.src), "s"(ldsa) : "memory");
#pragma unroll
    for (int i = 0; i < 8; ++i) LD16_LDS(ldsa + i * 1024, voff, rs, un.src + i * 8192);
}

__device__ __forceinline__ void unit_store(const lds_char* buf, u32x4 rd, const Unit& un, int lane) {
    const u32 voff = (u32)((lane >> 3) * 2048 + (lane & 7) * 16);
    const u32 ds = un.isw ? 256u : 128u;
    const bool isw = un.isw != 0;
    const int roff = isw ? lane * 16 : (lane & 7) * 128 + (lane >> 3) * 16;
    u32x4 v[8];
#pragma unroll
    for (int i = 0; i < 8; ++i) v[i] = *(const __attribute__((address_space(3))) u32x4*)(buf + roff + i * 1024);
#pragma unroll
    for (int t = 0; t < 8; ++t) {
        u32x4 o;
#pragma unroll
        for (int m = 0; m < 4; ++m) {
            const u32 tr = __builtin_amdgcn_perm(v[2 * m + 1][t >> 1], v[2 * m][t >> 1], (t & 1) ? 0x07060302u : 0x05040100u);
            o[m] = isw ? tr : v[t][m];
        }
        ST16_SYS(o, voff, rd, un.dst + t * ds);
    }
}

#define LD16_LDS_SYS(ldsa, voff, desc, soff, imm) do { u32 keep_m0; \
    asm volatile("s_mov_b32 %0, m0\n\ts_mov_b32 m0, %1\n\ts_nop 0\n\t" \
                 "buffer_load_dwordx4 %2, %3, %4 offen offset:" #imm " sc0 sc1 lds\n\ts_mov_b32 m0, %0" \
                 : "=&s"(keep_m0) : "s"(ldsa), "v"(voff), "s"(desc), "s"(soff) : "memory"); } while (0)

__device__ __forceinline__ void step_map(int s, int d, int& sb, int& sm) {
    if ((s >> 2) != (d >> 2)) { sb = 5 + ((s - d) & 3); sm = 4; }
    else { sb = 17 + ((s - d - 1) & 3); sm = 3; }
}

__device__ __forceinline__ Unit export_unit_of(int e) {
    const int isw = e >> 9, h2 = (e >> 3) & 63, kc8 = e & 7;
    Unit un;
    un.isw = isw;
    un.src = isw ? (u32)((kc8 * 64 * NDIM + h2 * 64) * 2) : (u32)((h2 * 64 * KSH + kc8 * 64) * 2);
    un.dst = (isw ? EXP_W + (h2 & 1) * 128 : EXP_X + (h2 & 1) * 1024) + (u32)((kc8 * 32 + (h2 >> 1)) * IMG);
    un.flag = 0;
    return un;
}
__device__ __forceinline__ void export_unit(const Params& p, lds_char* lds, int e, u32 seq, int lane) {
    const int rank = p.rank, isw = e >> 9, h2 = (e >> 3) & 63, kc8 = e & 7;
    Unit un = export_unit_of(e);
    unit_store(lds, mk_desc(p.wsl, WS_USED), un, lane);
    asm volatile("s_waitcnt vmcnt(0)" ::: "memory");
    const int ccls = kc8 < 5 ? rank >> 2 : 1 - (rank >> 2), kcl = kc8 < 5 ? kc8 : kc8 - 5;
    const int q = h2 >> 4, sel = ccls ? q & 1 : q >> 1, lq = ccls ? q >> 1 : q & 1;
    const int b8 = (h2 >> 1) & 7, half = h2 & 1;
    const int blk = isw ? lq * 8 + b8 : ((b8 >> 2) == ccls ? 8 : 0) + lq * 4 + (b8 & 3);
    const u32 gflag = isw ? FLAG_B + (u32)((blk >> 1) * 512 + (blk & 1) * 8 + half * 4) : FLAG_A + (u32)(blk * 512 + half * 4);
#pragma unroll
    for (int o = 0; o < 2; ++o) {
        const int c = ccls * 4 + (isw ? o * 2 + sel : sel * 2 + o);
        const u32 fo = c == rank ? gflag + kcl * 16 : EFLAG + (u32)(rank * 2048 + (kcl * 64 + isw * 32 + lq * 16 + (h2 & 15)) * 4);
        const u32x4 rc = mk_desc(p.ws[c], WS_USED);
        asm volatile("s_nop 4" :: "s"(rc) : "memory");
        ST4_SYS(seq, lane == 0 ? fo : OOB, rc);
    }
}

struct PUnit { u32 src; u32 dst; u32 flag; u32 wait; };
template <int NK>
__device__ __forceinline__ PUnit pdecode(int u, int rank, int d, int sb, int sm) {
    PUnit r;
    if (NK == 3 && u >= 64 * NK) {
        const int v = u - 64 * NK, g = v >> 5, tl = (v >> 2) & 7, q = v & 3, off = (tl & 3) * 16 + 2 * g + (tl >> 2);
        const int tile = ((d >> 1) & 1) * 64 + (d & 1) * 8 + off, ptile = ((rank >> 1) & 1) * 64 + (rank & 1) * 8 + off;
        r.src = OFF_P + (u32)(ptile * 32768 + q * 8192);
        r.dst = OFF_P2 + (u32)(tile * 32768 + q * 8192);
        r.flag = PF2 + (u32)(tile * 16 + q * 4);
        r.wait = (u32)(v >> 2) * 2;
        return r;
    }
    int kc, idx, g = 0, keep = 0;
    bool isw;
    if (u < 24 * NK) { kc = u / 24; idx = u % 24; isw = idx >= 16; idx &= 15; }
    else if (u < 48 * NK) {
        int v = u - 24 * NK, sg = v / (4 * NK);
        v %= 4 * NK; g = 1 + (sg >> 1); kc = v >> 2; idx = (v & 2) * 2 + (sg & 1) * 2 + (v & 1); isw = true;
    }
    else {
        const int v = u - 48 * NK, r = v >> 3, pre = NK == 5 ? X_LATE0 : 0, hw = r < pre || r >= pre + NK;
        kc = hw ? (r < pre ? r : r - NK) : r - pre; idx = hw * 8 + (v & 7); isw = false; keep = 8;
    }
    const int half = idx & 1, s = sb + sm * kc, kc8 = kc + (NK == 5 ? 0 : 5);
    r.wait = OOB;
    if (isw) {
        const int cb = (idx >> 2) * 8 + g * 2 + ((idx >> 1) & 1);
        r.src = EXP_W + (u32)((kc8 * 32 + (gcol(rank, cb) >> 7)) * IMG + half * 8192);
        r.dst = OFF_B + (u32)((s * 16 + cb) * IMG + half * 8192);
        r.flag = FLAG_B + (u32)((cb >> 1) * 512 + s * 16 + (cb & 1) * 8 + half * 4);
    } else {
        const int rb = (idx >> 1) + keep;
        r.src = EXP_X + (u32)((kc8 * 32 + (grow(rank, rb) >> 7)) * IMG + half * 8192);
        r.dst = OFF_A + (u32)((s * 16 + rb) * IMG + half * 8192);
        r.flag = FLAG_A + (u32)(rb * 512 + s * 16 + half * 4);
    }
    return r;
}

__device__ __forceinline__ void pull8k(u32 ldsa, u32x4 rs, u32 src, int lane) {
    const u32 voff = (u32)lane * 16;
    ldsa = __builtin_amdgcn_readfirstlane(ldsa);
    src = __builtin_amdgcn_readfirstlane(src);
    asm volatile("s_nop 4" :: "s"(ldsa), "s"(src) : "memory");
    LD16_LDS_SYS(ldsa, voff, rs, src, 0); LD16_LDS_SYS(ldsa, voff, rs, src, 1024);
    LD16_LDS_SYS(ldsa, voff, rs, src, 2048); LD16_LDS_SYS(ldsa, voff, rs, src, 3072);
    LD16_LDS_SYS(ldsa + 4096, voff, rs, src + 4096, 0); LD16_LDS_SYS(ldsa + 4096, voff, rs, src + 4096, 1024);
    LD16_LDS_SYS(ldsa + 4096, voff, rs, src + 4096, 2048); LD16_LDS_SYS(ldsa + 4096, voff, rs, src + 4096, 3072);
}

__device__ __forceinline__ void push_unit(const Params& p, lds_char* lds, int d, int u, u32 seq, int lane, u64 t_start) {
    int sb, sm;
    step_map(p.rank, d, sb, sm);
    const Unit un = (d >> 2) == (p.rank >> 2) ? decode<5>(u, d, sb, sm) : decode<3>(u, d, sb, sm);
    const u32x4 rp = mk_desc(p.ws[d], WS_USED);
    unit_load(__builtin_amdgcn_readfirstlane((u32)(uintptr_t)lds), p, un, lane);
    asm volatile("s_waitcnt vmcnt(0)" ::: "memory");
    if (u >= NPUSH1)
        while (__builtin_amdgcn_s_memrealtime() - t_start < PUSH_DELAY) __builtin_amdgcn_s_sleep(1);
    unit_store(lds, rp, un, lane);
    asm volatile("s_waitcnt vmcnt(0)" ::: "memory");
    ST4_SYS(seq, lane == 0 ? un.flag : OOB, rp);
}

template <int NK>
__device__ __forceinline__ void mover(const Params& p, lds_char* lds, int d, int wv, u32 seq, int lane) {
    const int rank = p.rank;
    const __amdgpu_buffer_rsrc_t rw = mk_rsrc(p.wsl, WS_USED);
    const u32x4 rl = mk_desc(p.wsl, WS_USED), rp = mk_desc(p.ws[d], WS_USED);
    const u32 ldsa = __builtin_amdgcn_readfirstlane((u32)(uintptr_t)lds);
    int sb, sm;
    step_map(d, rank, sb, sm);
    {
        const u64 t0 = __builtin_amdgcn_s_memrealtime();
        for (int spins = 1;; ++spins) {
            bool ok = true;
#pragma unroll
            for (int i = 0; i < NK; ++i)
                ok &= __builtin_amdgcn_raw_buffer_load_b32(rw, EFLAG + (u32)(d * 2048 + i * 256 + lane * 4), 0, AUX_SYS) >= seq;
            if (__all(ok) || ((spins & 63) == 0 && timed_out(p, seq, t0))) break;
        }
    }
    const u32 pfo = PFLAG + (u32)((((d >> 1) & 1) * 64 + (d & 1) * 8 + ((lane >> 1) & 3) * 16 + 2 * (lane >> 4) + ((lane >> 3) & 1)) * 32 + (lane & 1) * 16);
    u64 pmask = 0;
    const u32 tbase = (seq - 1u) * (u32)(NU - NPUSH - NIW) - (u32)(NPUSH + 2 * NIW);
    int u = NPUSH + wv, u1 = NPUSH + NIW + wv;
    PUnit cur = pdecode<NK>(u, rank, d, sb, sm);
    u32 tk = (u32)__builtin_amdgcn_raw_ptr_buffer_atomic_add_i32(1, rw, lane == 0 ? TICK + d * 256 : OOB, 0, AUX_SC1);
    pull8k(ldsa, rp, cur.src, lane);
    u32x4 pf = {0u, 0u, 0u, 0u};
    u32 pflag = OOB;
    int par = 0;
    for (;;) {
        asm volatile("s_waitcnt vmcnt(0)" ::: "memory");
        const u32 n = __builtin_amdgcn_readfirstlane(tk) - tbase;
        if (NK == 3) pmask |= __ballot(pf[0] >= seq && pf[1] >= seq && pf[2] >= seq && pf[3] >= seq);
        asm volatile("" :: "s"(n), "s"(pmask) : "memory");
        const int u2 = (u1 < NU && n < (u32)NU) ? (int)n : NU;
        const PUnit nxt = pdecode<NK>(u1 < NU ? u1 : u, rank, d, sb, sm);
        const bool now = u1 < NU && !(NK == 3 && nxt.wait != OOB && ((pmask >> nxt.wait) & 3) != 3);
        if (now) {
            tk = (u32)__builtin_amdgcn_raw_ptr_buffer_atomic_add_i32(1, rw, (u2 < NU && lane == 0) ? TICK + d * 256 : OOB, 0, AUX_SC1);
            pull8k(ldsa + (par ^ 8192), rp, nxt.src, lane);
            if (NK == 3) pf = __builtin_amdgcn_raw_buffer_load_b128(rw, pfo, 0, AUX_SYS);
        }
        ST4_SYS(seq, pflag, rl);
        u32x4 v[8];
#pragma unroll
        for (int i = 0; i < 8; ++i) v[i] = *(const __attribute__((address_space(3))) u32x4*)(lds + par + lane * 16 + i * 1024);
#pragma unroll
        for (int i = 0; i < 8; ++i) ST16_SYS(v[i], (u32)lane * 16, rl, cur.dst + i * 1024);
        pflag = lane == 0 ? cur.flag : OOB;
        if (!now) {
            asm volatile("s_waitcnt vmcnt(0)" ::: "memory");
            ST4_SYS(seq, pflag, rl);
            pflag = OOB;
            if (u1 >= NU) break;
            const u64 t0 = __builtin_amdgcn_s_memrealtime();
            for (;;) {
                const u32x4 f = __builtin_amdgcn_raw_buffer_load_b128(rw, pfo, 0, AUX_SYS);
                pmask = __ballot(f[0] >= seq && f[1] >= seq && f[2] >= seq && f[3] >= seq);
                if (((pmask >> nxt.wait) & 3) == 3 || timed_out(p, seq, t0)) break;
                __builtin_amdgcn_s_sleep(1);
            }
            tk = (u32)__builtin_amdgcn_raw_ptr_buffer_atomic_add_i32(1, rw, (u2 < NU && lane == 0) ? TICK + d * 256 : OOB, 0, AUX_SC1);
            pull8k(ldsa + (par ^ 8192), rp, nxt.src, lane);
        }
        cur = nxt; u = u1; u1 = u2; par ^= 8192;
    }
}

constexpr int OB_NT = NO / NWV, OB_NB = 6;
constexpr int OB_IMG = 0, OB_MASK = 8 * 2304, OB_IN = OB_MASK + OB_NT * 2048, OB_PITCH = 8208,
              OB_LUT = OB_IN + 2 * OB_NB * OB_PITCH, OB_INFO = OB_LUT + 64, OB_LEN = OB_INFO + 64, OB_BASE = OB_LEN + 128,
              OB_END = OB_BASE + 1024;
typedef volatile __attribute__((address_space(3))) u32 lds_vu32;
__device__ __forceinline__ int tile_of(int wv, int i) { return i * 8 + ((wv + (i & 14)) & 7); }
__device__ __forceinline__ u32 plane_bytes(u32 n) { return ((n + 7u) / 8u + 15u) & ~15u; }
#define WG_BARRIER() do { asm volatile("s_waitcnt lgkmcnt(0)" ::: "memory"); __builtin_amdgcn_s_barrier(); asm volatile("" ::: "memory"); } while (0)

__device__ __forceinline__ void out_request(const Params& p, lds_char* lds, int d, int wv, u32 seq, int lane) {
    const int rank = p.rank;
    const __amdgpu_buffer_rsrc_t rw = mk_rsrc(p.wsl, WS_USED);
    const u32x4 rp = mk_desc(p.ws[d], WS_USED);
    const u32 ldsa = __builtin_amdgcn_readfirstlane((u32)(uintptr_t)lds);
    const u32 voff = (u32)lane * 16;
    u32 have = 0;
    u32 tm = 0, lens = 0;
    auto wait_streams = [&](u32 need) {
        const u64 t0 = __builtin_amdgcn_s_memrealtime();
        for (int spins = 1;; ++spins) {
            const u32x2 g = __builtin_amdgcn_raw_buffer_load_b64(rw, OFLAG + (u32)(d * 1024 + tile_of(wv, lane & 15) * 8), 0, AUX_SYS);
            lens = g[0];
            tm = __builtin_amdgcn_readfirstlane(__builtin_amdgcn_raw_buffer_load_b32(rw, TMO, 0, AUX_SC1));
            have = (u32)__ballot(g[1] == seq);
            if ((have & need) == need || tm == seq) break;
            if ((spins & 63) == 0 && timed_out(p, seq, t0)) { tm = seq; break; }
        }
    };
    for (u32 todo = 0xffffu; todo;) {
        if (!(have & todo)) {
            const u64 t0 = __builtin_amdgcn_s_memrealtime();
            for (int spins = 1; !(have & todo); ++spins) {
                have = (u32)__ballot(__builtin_amdgcn_raw_buffer_load_b32(rw, MFLAG + (u32)(d * 512 + tile_of(wv, lane & 15) * 4), 0, AUX_SYS) >= seq);
                if ((spins & 63) == 0 && timed_out(p, seq, t0)) have = 0xffffu;
            }
        }
        const int i = __builtin_ctz(have & todo);
        todo &= ~(1u << i);
        asm volatile("s_waitcnt vmcnt(6)" ::: "memory");
        const u32 src = __builtin_amdgcn_readfirstlane(OFF_M + (u32)tile_of(wv, i) * 2048);
        const u32 la = __builtin_amdgcn_readfirstlane(ldsa + OB_MASK + i * 2048);
        asm volatile("s_nop 4" :: "s"(la), "s"(src) : "memory");
        LD16_LDS_SYS(la, voff, rp, src, 0); LD16_LDS_SYS(la, voff, rp, src, 1024);
    }
    constexpr int NH = 2 * OB_NT;
    int rbuf = 0;
    auto request = [&](int i) {
        const u32 bytes = i < NH ? 7u * min(((u32)__builtin_amdgcn_readlane(lens, i >> 1) >> ((i & 1) * 16)) & 0xffffu, 1024u) : 0u;
        const u32 src = __builtin_amdgcn_readfirstlane(OFF_Q + (u32)(tile_of(wv, (i >> 1) & 15) * 2 + (i & 1)) * QPITCH);
        const u32 la = __builtin_amdgcn_readfirstlane(ldsa + OB_IN + rbuf * OB_PITCH);
        rbuf = rbuf == 2 * OB_NB - 1 ? 0 : rbuf + 1;
        asm volatile("s_nop 4" :: "s"(la), "s"(src) : "memory");
        LD16_LDS_SYS(la, voff < bytes ? voff : OOB, rp, src, 0);
        LD16_LDS_SYS(la, 1024 + voff < bytes ? voff : OOB, rp, src, 1024);
        LD16_LDS_SYS(la, 2048 + voff < bytes ? voff : OOB, rp, src, 2048);
        LD16_LDS_SYS(la, 3072 + voff < bytes ? voff : OOB, rp, src, 3072);
        LD16_LDS_SYS(la + 4096, 4096 + voff < bytes ? voff : OOB, rp, src + 4096, 0);
        LD16_LDS_SYS(la + 4096, 5120 + voff < bytes ? voff : OOB, rp, src + 4096, 1024);
        LD16_LDS_SYS(la + 4096, 6144 + voff < bytes ? voff : OOB, rp, src + 4096, 2048);
    };
    wait_streams(1u);
    asm volatile("s_waitcnt vmcnt(0)" ::: "memory");
    request(0); request(1);
    __builtin_amdgcn_s_barrier();
    if ((have & 0xffffu) != 0xffffu) wait_streams(0xffffu);
    if (lane == 0) *(lds_vu32*)(lds + OB_INFO) = tm == seq;
    for (int i = 2; i < ODEPTH; ++i) request(i);
    for (int i = 0; i < NH; ++i) {
        asm volatile("s_waitcnt vmcnt(%0)" :: "n"(7 * (ODEPTH - 1)) : "memory");
        request(i + ODEPTH);
        if (i & 1) WG_BARRIER();
    }
    asm volatile("s_waitcnt vmcnt(0)" ::: "memory");
}

__device__ __forceinline__ void out_restore(lds_char* lds, int h, int lane) {
    WG_BARRIER();
    for (int i = 0, buf = h; i < OB_NT; ++i, buf = buf >= 2 * OB_NB - 2 ? h : buf + 2) {
        WG_BARRIER();
        const u32 L = __builtin_amdgcn_readfirstlane(*(lds_vu32*)(lds + OB_LEN + (2 * i + h) * 4));
        if ((u32)lane * 16 < L) {
            const lds_char* pl = lds + OB_IN + buf * OB_PITCH + lane * 16;
            u32x4 p7 = {0u, 0u, 0u, 0u};
#pragma unroll
            for (int j = 0; j < 7; ++j)
                p7 |= ((*(const __attribute__((address_space(3))) u32x4*)(pl + j * L) >> 7) & 0x01010101u) << j;
            *(__attribute__((address_space(3))) u32x4*)(pl + 7 * L) = p7;
        }
    }
    WG_BARRIER();
}

__device__ __forceinline__ void out_expand(const Params& p, lds_char* lds, int d, int wv, int e, int lane) {
    const int k = e & 3, h = e >> 2;
    const u32x4 ro = mk_desc(p.out[p.rank], OUT_BYTES);
    if (e == 0 && lane < 16) {
        u32 sel = 0, c = 0;
        for (int i = 0; i < 4; ++i) { sel |= (((lane >> i) & 1) ? c++ : 0x0cu) << (8 * i); }
        *(__attribute__((address_space(3))) u32*)(lds + OB_LUT + lane * 4) = sel;
    }
    WG_BARRIER();
#pragma unroll
    for (int q = 0; q < 4; ++q) {
        const int i = 4 * e + q;
        const u32x4 m = *(const __attribute__((address_space(3))) u32x4*)(lds + OB_MASK + i * 1024 + lane * 16);
        u32 c = 0;
#pragma unroll
        for (int x = 0; x < 4; ++x) c += __builtin_popcount(m[x] & 0xffffu) + (__builtin_popcount(m[x] >> 16) << 16);
        c += (u32)__builtin_amdgcn_update_dpp(0, (int)c, 0x111, 0xf, 0xf, true);
        c += (u32)__builtin_amdgcn_update_dpp(0, (int)c, 0x112, 0xf, 0xf, true);
        c += (u32)__builtin_amdgcn_update_dpp(0, (int)c, 0x114, 0xf, 0xf, true);
        c += (u32)__builtin_amdgcn_update_dpp(0, (int)c, 0x118, 0xf, 0xf, true);
        u32 bs[9];
        bs[0] = 0;
#pragma unroll
        for (int x = 0; x < 4; ++x) {
            const u32 t = __builtin_amdgcn_readlane(c, 16 * x + 15);
            bs[2 * x + 1] = bs[2 * x] + (t & 0xffffu);
            bs[2 * x + 2] = bs[2 * x + 1] + (t >> 16);
        }
        if (lane == 0) {
            *(__attribute__((address_space(3))) u32x4*)(lds + OB_BASE + i * 32) = u32x4{bs[0], bs[1], bs[2], bs[3]};
            *(__attribute__((address_space(3))) u32x4*)(lds + OB_BASE + i * 32 + 16) = u32x4{bs[4], bs[5], bs[6], bs[7]};
            *(__attribute__((address_space(3))) u32*)(lds + OB_LEN + i * 4) = plane_bytes(bs[8]);
        }
    }
    WG_BARRIER();
    lds_char* img = lds + OB_IMG + e * 2304;
    const int fr = lane & 15, fq = lane >> 4, sh = (k & 1) * 16;
    const u32 ovoff = (u32)((e * 16 + (lane >> 3)) * NDIM + (lane & 7) * 16);
    for (int i = 0, buf = h; i < OB_NT; ++i, buf = buf >= 2 * OB_NB - 2 ? h : buf + 2) {
        u32 pos[2][4], sel[2][4];
#pragma unroll
        for (int wn = 0; wn < 2; ++wn) {
            const int pw = (k >> 1) * 2 + wn;
            const u32 m = *(const __attribute__((address_space(3))) u32*)(lds + OB_MASK + i * 2048 + h * 1024 + pw * 256 + lane * 4);
            u32 run = *(const __attribute__((address_space(3))) u32*)(lds + OB_BASE + (i * 2 + h) * 32 + pw * 8 + (k & 1) * 4);
            nibble_positions(m >> sh, run, pos[wn]);
#pragma unroll
            for (int j = 0; j < 4; ++j)
                sel[wn][j] = *(const __attribute__((address_space(3))) u32*)(lds + OB_LUT + ((m >> (sh + 4 * j)) & 15u) * 4);
        }
        WG_BARRIER();
        const lds_char* in = lds + OB_IN + buf * OB_PITCH;
        const bool bad = *(lds_vu32*)(lds + OB_INFO) != 0;
        u32x2 o[4];
#pragma unroll
        for (int j = 0; j < 4; ++j)
#pragma unroll
            for (int wn = 0; wn < 2; ++wn) {
                const __attribute__((address_space(3))) u32* sp = (const __attribute__((address_space(3))) u32*)(in + (pos[wn][j] & ~3u));
                o[j][wn] = __builtin_amdgcn_perm(0u, __builtin_amdgcn_alignbyte(sp[1], sp[0], pos[wn][j] & 3u), sel[wn][j]) & 0x7f7f7f7fu;
            }
#pragma unroll
        for (int j = 0; j < 4; ++j) *(__attribute__((address_space(3))) u32x2*)(img + (fq * 4 + j) * 144 + fr * 8) = o[j];
        const int tile = tile_of(wv, i);
        const u32 obase = (u32)(grow(d, 8 + (tile >> 4)) * NDIM + gcol(d, tile & 15));
        u32x4 v[2];
#pragma unroll
        for (int r = 0; r < 2; ++r) {
            v[r] = *(const __attribute__((address_space(3))) u32x4*)(img + (r * 8 + (lane >> 3)) * 144 + (lane & 7) * 16);
            if (bad) v[r] = u32x4{0x80808080u, 0x80808080u, 0x80808080u, 0x80808080u};
        }
#pragma unroll
        for (int r = 0; r < 2; ++r) ST16_SYS(v[r], ovoff, ro, obase + r * 8 * NDIM);
    }
}

typedef volatile __attribute__((address_space(3))) int lds_int;
constexpr int SLOT = 2 * IMG;

__device__ __forceinline__ void glds4k(const char* g, lds_char* l) {
    const __attribute__((address_space(1))) void* gp = (const __attribute__((address_space(1))) void*)g;
    __attribute__((address_space(3))) void* lp = (__attribute__((address_space(3))) void*)l;
    __builtin_amdgcn_global_load_lds(gp, lp, 16, 0, 0);
    __builtin_amdgcn_global_load_lds(gp, lp, 16, 1024, 0);
    __builtin_amdgcn_global_load_lds(gp, lp, 16, 2048, 0);
    __builtin_amdgcn_global_load_lds(gp, lp, 16, 3072, 0);
}

__device__ __forceinline__ void gemm_poller(const Params& p, lds_char* lds, int rb, int cb, u32 seq, int lane) {
    const bool keep = rb >= 8;
    const __amdgpu_buffer_rsrc_t rw = mk_rsrc(p.wsl, WS_USED);
    const u32 foff = (lane < 32 ? FLAG_A + rb * 512 : FLAG_B + (cb >> 1) * 512 + (cb & 1) * 8) + (lane & 31) * 16;
    lds_int* lready = (lds_int*)(lds + LDS_CTL) + keep;
    lds_int* order = (lds_int*)(lds + LDS_ORD) + keep * 32;
    u32 listed = 0;
    const char* psrc = p.wsl + OFF_P2 + ((rb & 7) * 16 + cb) * 32768 + lane * 16;
    const u32 pfoff = PF2 + ((rb & 7) * 16 + cb) * 16 + (lane & 3) * 4;
    bool pdone = !keep;
    auto poll = [&]() -> u32 {
        const u32x2 f = __builtin_amdgcn_raw_buffer_load_b64(rw, foff, 0, AUX_SYS);
        const u64 m = __ballot(f[0] >= seq && f[1] >= seq);
        return (u32)m & (u32)(m >> 32);
    };
    u32 pfv = 0;
    auto pload = [&]() { pfv = __builtin_amdgcn_raw_buffer_load_b32(rw, pdone ? OOB : pfoff, 0, AUX_SYS); };
    auto pstep = [&]() {
        if (!pdone && __all(pfv >= seq)) {
#pragma unroll
            for (int i = 0; i < 8; ++i) glds4k(psrc + i * 4096, lds + LDS_P + i * 4096);
            pdone = true;
        }
    };
    int ready = 0;
    for (int t = 0; t < NSTEP; ++t) {
        if (ready < NSTEP && ready < t + NBUF) {
            const u64 t0 = __builtin_amdgcn_s_memrealtime();
            for (int spins = 1;; ++spins) {
                pload();
                u32 fresh = poll() & ~listed;
                if (!fresh && ready <= t && (spins & 63) == 0 && timed_out(p, seq, t0)) fresh = ~listed;
                if (fresh) {
                    if (lane < 32 && ((fresh >> lane) & 1u)) order[ready + __builtin_popcount(fresh & ((1u << lane) - 1u))] = lane;
                    listed |= fresh;
                    ready += __builtin_popcount(fresh);
                    if (lane == 0) lready[0] = ready;
                }
                if (ready > t + 1 || ready == NSTEP) break;
                if (ready > t && __builtin_amdgcn_readfirstlane(min(lready[2], lready[4])) > t) break;
                pstep();
            }
            asm volatile("s_waitcnt lgkmcnt(0)" ::: "memory");
        }
        __builtin_amdgcn_s_barrier();
    }
    if (!pdone) {
        const u64 t0 = __builtin_amdgcn_s_memrealtime();
        for (int spins = 1;; ++spins) {
            pload();
            pstep();
            if (pdone || ((spins & 63) == 0 && timed_out(p, seq, t0))) break;
        }
    }
    asm volatile("s_waitcnt vmcnt(0)" ::: "memory");
    __builtin_amdgcn_s_barrier();
}

__device__ __forceinline__ void gemm_loader(const Params& p, lds_char* lds, int rb, int cb, int lw, int lane) {
    const bool keep = rb >= 8;
    const int lp = lw >> 1, lh = lw & 1;
    lds_int* order = (lds_int*)(lds + LDS_ORD) + keep * 32;
    const char* wsl = p.wsl;
    lds_int* lready = (lds_int*)(lds + LDS_CTL) + keep;
    const char* ga = wsl + OFF_A + rb * IMG + lh * 8192 + lane * 16;
    const char* gb = wsl + OFF_B + cb * IMG + lh * 8192 + lane * 16;
    const char* ea = wsl + EXP_X + (grow(p.rank, rb) >> 7) * IMG + lh * 8192 + lane * 16;
    const char* eb = wsl + EXP_W + (gcol(p.rank, cb) >> 7) * IMG + lh * 8192 + lane * 16;
    int next = lp, ready = 0;
    auto try_issue = [&](int lim) {
        if (next >= lim || next >= NSTEP) return;
        if (ready <= next) ready = __builtin_amdgcn_readfirstlane(lready[0]);
        if (ready <= next) return;
        const int sl = __builtin_amdgcn_readfirstlane(order[next]) & 31;
        lds_char* dst = lds + (next % NBUF) * SLOT + lh * 8192;
        const bool own = sl < 5;
        const size_t so = (size_t)sl * (own ? 32 * IMG : 16 * IMG);
        const char* sa = (own ? ea : ga) + so;
        const char* sb = (own ? eb : gb) + so;
        glds4k(sa, dst); glds4k(sa + 4096, dst + 4096);
        glds4k(sb, dst + IMG); glds4k(sb + 4096, dst + IMG + 4096);
        next += 2;
    };
    for (int t = 0; t < NSTEP; ++t) {
        if ((t & 1) == lp) {
            while (next <= t) {
                try_issue(t + 1);
                if (next <= t) __builtin_amdgcn_s_sleep(1);
            }
            asm volatile("s_waitcnt vmcnt(0)" ::: "memory");
            if (lane == 0) lready[2 + 2 * lh] = t + 1;
        } else {
            while (next == t + 1 && next < NSTEP) {
                try_issue(t + 2);
                if (next == t + 1 && __builtin_amdgcn_readfirstlane(min(lready[2], lready[4])) > t) break;
            }
        }
        WG_BARRIER();
        try_issue(t + NBUF);
    }
    __builtin_amdgcn_s_barrier();
}

__device__ __forceinline__ void frag_mma(f32x4 (&acc)[2][4], const bf16x8 (&fa)[2], const bf16x8 (&fb)[4]) {
#pragma unroll
    for (int m = 0; m < 2; ++m)
#pragma unroll
        for (int n = 0; n < 4; ++n)
            acc[m][n] = __builtin_amdgcn_mfma_f32_16x16x32_bf16(fa[m], fb[n], acc[m][n], 0, 0, 0);
}
__device__ __forceinline__ void frag_read(bf16x8 (&fa)[2], bf16x8 (&fb)[4], const lds_char* a, const lds_char* b) {
#pragma unroll
    for (int m = 0; m < 2; ++m) fa[m] = *(const __attribute__((address_space(3))) bf16x8*)(a + m * 256);
#pragma unroll
    for (int n = 0; n < 4; ++n) fb[n] = *(const __attribute__((address_space(3))) bf16x8*)(b + n * 256);
}
__device__ __forceinline__ void interleave() {
#pragma unroll
    for (int i = 0; i < 6; ++i) {
        __builtin_amdgcn_sched_group_barrier(0x100, 1, 0);
        __builtin_amdgcn_sched_group_barrier(0x8, 1, 0);
    }
    __builtin_amdgcn_sched_barrier(0);
}

__device__ __forceinline__ void gemm_compute(lds_char* lds, int wave, int lane, f32x4 (&acc)[2][4]) {
    const int fr = lane & 15, fq = lane >> 4;
    const int aoff = fq * 2048 + ((wave >> 1) * 32 + fr) * 16;
    const int boff = IMG + fq * 2048 + (wave & 1) * 1024 + fr * 16;
    bf16x8 a0[2], b0[4], a1[2], b1[4];
#pragma unroll
    for (int m = 0; m < 2; ++m) {
#pragma unroll
        for (int n = 0; n < 4; ++n) acc[m][n] = f32x4{0.f, 0.f, 0.f, 0.f};
        a1[m] = bf16x8{0, 0, 0, 0, 0, 0, 0, 0};
    }
#pragma unroll
    for (int i = 0; i < 4; ++i) b1[i] = bf16x8{0, 0, 0, 0, 0, 0, 0, 0};
    for (int t = 0; t < NSTEP; ++t) {
        asm volatile("s_waitcnt lgkmcnt(0)" ::: "memory");
        __builtin_amdgcn_s_barrier();
        asm volatile("" ::: "memory");
        const lds_char* buf = lds + (t % NBUF) * SLOT;
        __builtin_amdgcn_sched_barrier(0);
        frag_read(a0, b0, buf + aoff, buf + boff);
        frag_mma(acc, a1, b1);
        interleave();
        frag_read(a1, b1, buf + aoff + 8192, buf + boff + 8192);
        frag_mma(acc, a0, b0);
        interleave();
    }
    frag_mma(acc, a1, b1);
    asm volatile("s_waitcnt lgkmcnt(0)" ::: "memory");
    __builtin_amdgcn_s_barrier();
    asm volatile("" ::: "memory");
}

__device__ __forceinline__ u32 pack_bf16(float lo, float hi) {
    typedef __attribute__((ext_vector_type(2))) __bf16 bf2;
    bf2 r; r[0] = (__bf16)lo; r[1] = (__bf16)hi;
    return __builtin_bit_cast(u32, r);
}

__device__ __forceinline__ u32x4 pack8(const f32x4& a, const f32x4& b) {
    u32x4 o;
    o[0] = pack_bf16(a[0], a[1]); o[1] = pack_bf16(a[2], a[3]);
    o[2] = pack_bf16(b[0], b[1]); o[3] = pack_bf16(b[2], b[3]);
    return o;
}

static_assert(OB_END <= LDS_BYTES && LDS_ORD + 256 <= LDS_BYTES, "LDS map");

__global__ void __launch_bounds__(THREADS, 4) fused_kernel(Params p) {
    extern __shared__ __attribute__((aligned(16))) char smem[];
    lds_char* lds = (lds_char*)smem;
    const int tid = threadIdx.x, lane = tid & 63;
    const int wave = __builtin_amdgcn_readfirstlane(tid >> 6);
    const int rank = p.rank;
    const int xcd = blockIdx.x & 7, slot = blockIdx.x >> 3;
    char* wsl = p.wsl;
    const u64 t_start = __builtin_amdgcn_s_memrealtime();
    const u32 cntv = __hip_atomic_load((u32*)(wsl + CNT), __ATOMIC_RELAXED, __HIP_MEMORY_SCOPE_AGENT);
    const int rbk = (xcd & 1) * 4 + (slot & 3);
    const int cb = (xcd >> 1) * 4 + ((slot >> 2) & 3);
    const int tile = rbk * 16 + cb;
    if (slot >= 16 && wave < 8)
        unit_load(__builtin_amdgcn_readfirstlane((u32)(uintptr_t)(lds + 4 * IMG + wave * 8192)), p, export_unit_of(wave * 128 + tile), lane);
    asm volatile("s_waitcnt vmcnt(0)" ::: "memory");
    const u32 seq = __builtin_amdgcn_readfirstlane(cntv) + 1u;

    if (slot < 8) {
        const int d = (rank + slot) & 7, wv = (xcd + slot) & 7;
        if (slot == 0) {
            if (wave == 0 && xcd) {
                const int d9 = (rank + xcd) & 7;
                if ((d9 >> 2) == (rank >> 2)) mover<5>(p, lds, d9, 8, seq, lane);
                else mover<3>(p, lds, d9, 8, seq, lane);
            }
            return;
        }
        if (wave > 10) return;
        if (wave > 8) { out_restore(lds, wave - 9, lane); return; }
        if (wave) { out_expand(p, lds, d, wv, wave - 1, lane); return; }
        if ((d >> 2) == (rank >> 2)) mover<5>(p, lds, d, wv, seq, lane);
        else mover<3>(p, lds, d, wv, seq, lane);
        out_request(p, lds, d, wv, seq, lane);
        return;
    }

    if (wave == 8) {
        if (lane < 6) ((lds_int*)(lds + LDS_CTL))[lane] = 0;
    }
    WG_BARRIER();
    const int srb = (xcd & 1) * 4 + (slot & 3);
    const int scb = (xcd >> 2) * 8 + (slot < 16 ? 4 : 0) + ((slot >> 2) & 1) * 2 + ((xcd >> 1) & 1);
    const int first = slot < 24 ? 0 : 1, last = slot < 16 ? 0 : 1;
    if (wave >= 8) {
        for (int pass = first; pass <= last; ++pass) {
            const int rb = pass ? 8 + rbk : srb, c = pass ? cb : scb;
            if (wave == 12) gemm_poller(p, lds, rb, c, seq, lane);
            else gemm_loader(p, lds, rb, c, wave - 8, lane);
        }
        return;
    }
    if (slot >= 16) export_unit(p, lds + 4 * IMG + wave * 8192, wave * 128 + tile, seq, lane);
    else if (wave * 64 + (slot - 8) * 8 + xcd < 7 * NPUSH) {
        const int id = wave * 64 + (slot - 8) * 8 + xcd;
        push_unit(p, lds + 4 * IMG + wave * 8192, (rank + 1 + id % 7) & 7, id / 7, seq, lane, t_start);
    }
    const __amdgpu_buffer_rsrc_t rw = mk_rsrc(wsl, WS_USED);
    f32x4 acc[2][4];
    for (int pass = first;; ++pass) {
        gemm_compute(lds, wave, lane, acc);
        if (pass) break;
        int partner, ptile;
        pair_of(rank, srb, scb, partner, ptile);
        const __amdgpu_buffer_rsrc_t rd = mk_rsrc(p.ws[partner], WS_USED);
#pragma unroll
        for (int m = 0; m < 2; ++m)
#pragma unroll
            for (int h = 0; h < 2; ++h)
                __builtin_amdgcn_raw_buffer_store_b128(pack8(acc[m][2 * h], acc[m][2 * h + 1]), rw, tid * 16,
                                                       OFF_P + (srb * 16 + scb) * 32768 + (m * 2 + h) * 8192, AUX_SYS);
        asm volatile("s_waitcnt vmcnt(0)" ::: "memory");
        if (lane == 0) __builtin_amdgcn_raw_buffer_store_b32(seq, rd, PFLAG + ptile * 32 + wave * 4, 0, AUX_SYS);
        if (!last) return;
    }
    float mx = 0.f;
#pragma unroll
    for (int m = 0; m < 2; ++m)
#pragma unroll
        for (int h = 0; h < 2; ++h) {
            const u32x4 o = *(const __attribute__((address_space(3))) u32x4*)(lds + LDS_P + (m * 2 + h) * 8192 + tid * 16);
#pragma unroll
            for (int q = 0; q < 4; ++q) {
                float lo = __builtin_bit_cast(float, o[q] << 16), hi = __builtin_bit_cast(float, o[q] & 0xffff0000u);
                f32x4& a = acc[m][2 * h + (q >> 1)];
                a[(q & 1) * 2] = fmaxf(a[(q & 1) * 2] + lo, 0.f);
                a[(q & 1) * 2 + 1] = fmaxf(a[(q & 1) * 2 + 1] + hi, 0.f);
                mx = fmaxf(mx, fmaxf(a[(q & 1) * 2], a[(q & 1) * 2 + 1]));
            }
        }
    u32 mask = 0;
    mx = wave_max(mx);
    volatile __attribute__((address_space(3))) float* lmax = (volatile __attribute__((address_space(3))) float*)(lds + LDS_CTL + 64);
    lds_vu32* lcnt = (lds_vu32*)(lds + LDS_CTL + 128);
    if (lane == 0) lmax[wave] = mx;
    WG_BARRIER();
#pragma unroll
    for (int j = 0; j < 8; ++j) mx = fmaxf(mx, lmax[j]);
    u32 L, L01;
    f32x4 col[8];
    {
        const __amdgpu_buffer_rsrc_t rd = mk_rsrc(p.ws[wave], WS_USED);
        u32x2 gr; gr[0] = __float_as_uint(mx); gr[1] = seq;
        if (lane == 0) __builtin_amdgcn_raw_buffer_store_b64(gr, rd, AMAX + (rank * 128 + tile) * 8, 0, AUX_SYS);
        asm volatile("" ::: "memory");
#pragma unroll
        for (int m = 0; m < 2; ++m)
#pragma unroll
            for (int j = 0; j < 4; ++j)
#pragma unroll
                for (int n = 0; n < 4; ++n) mask |= (acc[m][n][j] > 0.f ? 1u : 0u) << ((m * 4 + j) * 4 + n);
        __builtin_amdgcn_raw_buffer_store_b32(mask, rw, tid * 4, OFF_M + tile * 2048, AUX_SYS);
        u32 pos[2][4], wtot = 0;
        nibble_positions(mask, wtot, pos[0]);
        nibble_positions(mask >> 16, wtot, pos[1]);
        if (lane == 0) lcnt[wave] = wtot;
        asm volatile("s_waitcnt vmcnt(0)" ::: "memory");
        WG_BARRIER();
        if (lane == 0) __builtin_amdgcn_raw_buffer_store_b32(seq, rd, MFLAG + (rank * 128 + tile) * 4, 0, AUX_SYS);
        u32 base = 0, n[2] = {0u, 0u};
#pragma unroll
        for (int j = 0; j < 8; ++j) {
            const u32 c = __builtin_amdgcn_readfirstlane(lcnt[j]);
            if ((j >> 2) == (wave >> 2) && j < wave) base += c;
            n[j >> 2] += c;
        }
        L01 = plane_bytes(n[0]) | (plane_bytes(n[1]) << 16);
        L = (wave >> 2) ? L01 >> 16 : L01 & 0xffffu;
        lds_char* fs = lds + 32768 + (wave >> 2) * 32768;
#pragma unroll
        for (int dw = 0; dw < 8; ++dw) {
            u32 ps = (base + pos[dw >> 2][dw & 3]) * 4;
#pragma unroll
            for (int c = 0; c < 4; ++c) {
                const bool on = (mask >> (4 * dw + c)) & 1u;
                *(__attribute__((address_space(3))) float*)(on ? fs + ps : lds + 24576 + lane * 4) = acc[dw >> 2][c][dw & 3];
                ps += on ? 4u : 0u;
            }
        }
        WG_BARRIER();
#pragma unroll
        for (int j = 0; j < 8; ++j)
            col[j] = *(const __attribute__((address_space(3))) f32x4*)(fs + ((j * L + ((tid & 255) * 4 < L ? (tid & 255) * 4 : 0)) * 4));
        const u64 t0 = __builtin_amdgcn_s_memrealtime();
        u32x4 f;
        u32 tm;
        for (int spins = 1;; ++spins) {
            f = __builtin_amdgcn_raw_buffer_load_b128(rw, AMAX + wave * 1024 + lane * 16, 0, AUX_SYS);
            tm = __builtin_amdgcn_readfirstlane(__builtin_amdgcn_raw_buffer_load_b32(rw, TMO, 0, AUX_SC1));
            if (__all(f[1] == seq && f[3] == seq) || tm == seq) break;
            if ((spins & 63) == 0 && timed_out(p, seq, t0)) { tm = seq; break; }
            __builtin_amdgcn_s_sleep(2);
        }
        const u32 f0 = f[0], f2 = f[2];
        const float gm = wave_max(__uint_as_float(max(f0, f2)));
        if (lane == 0) { lmax[8 + wave] = gm; lcnt[8 + wave] = tm; }
    }
    __syncthreads();
    float amax = 0.f;
#pragma unroll
    for (int j = 0; j < 8; ++j) amax = fmaxf(amax, lmax[8 + j]);
    bool bad = false;
#pragma unroll
    for (int j = 0; j < 8; ++j) bad = bad || lcnt[8 + j] == seq;
    const float inv = amax > 0.f ? 1.0f / (amax / 127.0f) : 0.f;
    {
        u32 q[8];
#pragma unroll
        for (int j = 0; j < 8; ++j) {
            q[j] = 0;
#pragma unroll
            for (int c = 0; c < 4; ++c) q[j] = __builtin_amdgcn_cvt_pk_u8_f32(rintf(col[j][c] * inv), c, q[j]);
        }
        const u32 voff = (u32)(tid & 255) * 4 < L ? (u32)(tid & 255) * 4 : OOB;
#pragma unroll
        for (int j = 0; j < 7; ++j)
            __builtin_amdgcn_raw_buffer_store_b32((q[j] & 0x7f7f7f7fu) | (((q[7] >> j) & 0x01010101u) << 7), rw, voff,
                                                  OFF_Q + (u32)(tile * 2 + (wave >> 2)) * QPITCH + j * L, AUX_SYS);
    }
    {
        const int fr = lane & 15, fq = lane >> 4, wm = wave >> 1, wn = wave & 1;
#pragma unroll
        for (int m = 0; m < 2; ++m)
#pragma unroll
            for (int j = 0; j < 4; ++j) {
                u32 pk = 0;
#pragma unroll
                for (int n = 0; n < 4; ++n) pk = __builtin_amdgcn_cvt_pk_u8_f32(rintf(acc[m][n][j] * inv), n, pk);
                if (bad) pk = 0x80808080u;
                const int row = wm * 32 + m * 16 + fq * 4 + j;
                *(__attribute__((address_space(3))) u32*)(lds + row * 144 + fr * 8 + wn * 4) = pk;
            }
    }
    asm volatile("s_waitcnt vmcnt(0)" ::: "memory");
    WG_BARRIER();
    if (lane == 0) {
        const __amdgpu_buffer_rsrc_t rd = mk_rsrc(p.ws[wave], WS_USED);
        __builtin_amdgcn_raw_buffer_store_b64(u32x2{L01, seq}, rd, OFLAG + (rank * 128 + tile) * 8, 0, AUX_SYS);
        if (tile == 0 && wave == 0) __hip_atomic_store((u32*)(wsl + CNT), seq, __ATOMIC_RELAXED, __HIP_MEMORY_SCOPE_AGENT);
    }
    const __amdgpu_buffer_rsrc_t ro = mk_rsrc(p.out[rank], OUT_BYTES);
    const int r0 = wave * 16 + (lane >> 3), c0 = (lane & 7) * 16;
    const u32 voff = (u32)((grow(rank, 8 + rbk) + r0) * NDIM + gcol(rank, cb) + c0);
#pragma unroll
    for (int i = 0; i < 2; ++i)
        __builtin_amdgcn_raw_buffer_store_b128(*(const __attribute__((address_space(3))) u32x4*)(lds + (r0 + i * 8) * 144 + c0),
                                               ro, voff, i * 8 * NDIM, AUX_SYS);
}

extern "C" void kernel_launch(int rank, int nranks, ncclComm_t comm,
                              hipStream_t stream,
                              void* const* d_in, const int* in_sizes, int n_in,
                              void* d_out, int out_size,
                              const int* in_dtypes, int out_dtype,
                              void* const* d_out_all,
                              void* const* d_ws_all, size_t ws_bytes) {
    (void)comm; (void)in_dtypes; (void)out_dtype; (void)d_out;
    if (nranks != NR || (unsigned)rank >= (unsigned)NR || n_in < 2 || in_sizes[0] != MDIM * KSH ||
        in_sizes[1] != KSH * NDIM || out_size != MDIM * NDIM || ws_bytes < (size_t)WS_USED) {
        fprintf(stderr, "kernel_launch: unexpected problem shape\n");
        return;
    }
    static bool attr_set = false;
    if (!attr_set) {
        HIP_CHECK(hipFuncSetAttribute((const void*)fused_kernel, hipFuncAttributeMaxDynamicSharedMemorySize, LDS_BYTES));
        attr_set = true;
    }
    Params p;
    for (int r = 0; r < NR; ++r) { p.ws[r] = (char*)d_ws_all[r]; p.out[r] = (char*)d_out_all[r]; }
    p.x = (const char*)d_in[0];
    p.w = (const char*)d_in[1];
    p.rank = rank;
    p.wsl = (char*)d_ws_all[rank];
    hipLaunchKernelGGL(fused_kernel, dim3(GRID), dim3(THREADS), LDS_BYTES, stream, p);
    HIP_CHECK(hipGetLastError());
}
```
